```python
import numpy as np

kernel_name = "gpu_gemm_a2a_m4096_k4096_n4096_f32_reluq8"


ABSMAX_RTOL = 0.02

COMPUTE_DTYPE = "float32"
EPILOGUE = "relu_quant_int8"
IN_QUANT = None
OUT_QUANT = "int8"
IN_DTYPES = {"x": "float32", "w_mat": "float32"}
WORLD_SIZE = 8

def setup_inputs():

    x = np.random.randn(4096, 4096).astype(np.float32)
    w_mat = (np.random.randn(4096, 4096) / np.sqrt(4096)).astype(np.float32)
    return {"x": x, "w_mat": w_mat}

def _quant_dequant(y, amax):
    scale = amax / 127.0
    q = np.clip(np.round(y / scale), -127.0, 127.0)
    return (q * scale).astype(np.float32)
def _epilogue(y):
    y = np.maximum(y, 0.0)
    return _quant_dequant(y, np.abs(y).max())

def reference(x, w_mat):
    y = (x @ w_mat).astype(np.float32)
    return _epilogue(y)
```

```cpp
#include <hip/hip_runtime.h>
#include <rccl/rccl.h>
#include <cstdio>
#include <cstdint>

#define HIP_CHECK(expr) do { hipError_t _e = (expr); if (_e != hipSuccess) { \
    fprintf(stderr, "HIP error %s at %s:%d\n", hipGetErrorString(_e), __FILE__, __LINE__); \
    return; } } while (0)

typedef unsigned u32x4 __attribute__((ext_vector_type(4)));
typedef unsigned u32x2 __attribute__((ext_vector_type(2)));
typedef float f32x4 __attribute__((ext_vector_type(4)));
typedef float f32x16 __attribute__((ext_vector_type(16)));
typedef short bf16x8 __attribute__((ext_vector_type(8)));
typedef short bf16x4 __attribute__((ext_vector_type(4)));
typedef __bf16 bf16x2_t __attribute__((ext_vector_type(2)));
typedef float f32x2 __attribute__((ext_vector_type(2)));

constexpr int NR = 8;
constexpr int MS = 512;
constexpr int KD = 4096, ND = 4096;
constexpr int NS = 512;
constexpr int NWG = 256;
constexpr int TM = 128, TN = 64, BK = 64;
constexpr unsigned long long DEADLINE = 40000000ull;

constexpr size_t WS_CNT  = 0;
constexpr size_t WS_LMAX = WS_CNT + 256 * 128;
constexpr size_t WS_GMAX = WS_LMAX + 2 * 256 * 8;
constexpr size_t WS_DONE = WS_GMAX + 2 * 8 * 128;
constexpr size_t WS_TMO  = WS_DONE + 8 * 128;
constexpr size_t WS_END  = WS_TMO + 128;

struct Params {
  const float* x; const float* w;
  char* ws[NR]; signed char* out[NR];
  int rank; int pad;
};

__device__ __forceinline__ unsigned cvt_pk(float lo, float hi) {
  f32x2 v = {lo, hi};
  bf16x2_t b = __builtin_convertvector(v, bf16x2_t);
  return __builtin_bit_cast(unsigned, b);
}
__device__ __forceinline__ unsigned long long ld_sys64(const void* p) { return __hip_atomic_load((const unsigned long long*)p, __ATOMIC_RELAXED, __HIP_MEMORY_SCOPE_SYSTEM); }
__device__ __forceinline__ unsigned ld_sys32(const void* p) { return __hip_atomic_load((const unsigned*)p, __ATOMIC_RELAXED, __HIP_MEMORY_SCOPE_SYSTEM); }
__device__ __forceinline__ void st_sys64(void* p, unsigned long long v) { __hip_atomic_store((unsigned long long*)p, v, __ATOMIC_RELAXED, __HIP_MEMORY_SCOPE_SYSTEM); }
__device__ __forceinline__ void st_sys32(void* p, unsigned v) { __hip_atomic_store((unsigned*)p, v, __ATOMIC_RELAXED, __HIP_MEMORY_SCOPE_SYSTEM); }

__global__ __launch_bounds__(256) void gemm_a2a(Params p) {
  __shared__ __attribute__((aligned(16))) char smem[2 * (TM * BK * 2 + BK * TN * 2) + 64];
  const int tid = threadIdx.x, lane = tid & 63, wid = tid >> 6;
  const int b = blockIdx.x;
  char* myws = p.ws[p.rank];
  unsigned call = 0;
  if (tid == 0) { call = ld_sys32(myws + WS_CNT + b * 128) + 1; st_sys32(myws + WS_CNT + b * 128, call); }

  const int xcd = b & 7, j = b >> 3;
  const int tm = j & 3, tn = xcd * 8 + (j >> 2);
  const int wr = wid >> 1, wc = wid & 1;
  const float* xa = p.x + (size_t)(tm * TM) * KD;
  const float* wb = p.w + tn * TN;
  f32x4 ra[8], rb[4];
  auto gload = [&](int kt) {
#pragma unroll
    for (int i = 0; i < 8; ++i) ra[i] = *(const f32x4*)(xa + (size_t)(i * 16 + (tid >> 4)) * KD + kt * BK + (tid & 15) * 4);
#pragma unroll
    for (int i = 0; i < 4; ++i) rb[i] = *(const f32x4*)(wb + (size_t)(kt * BK + i * 16 + (tid >> 4)) * ND + (tid & 15) * 4);
  };
  auto lwrite = [&](int buf) {
    char* A = smem + buf * (TM * BK * 2 + BK * TN * 2); char* B = A + TM * BK * 2;
#pragma unroll
    for (int i = 0; i < 8; ++i) { int row = i * 16 + (tid >> 4), c8 = tid & 15;
      u32x2 v = {cvt_pk(ra[i][0], ra[i][1]), cvt_pk(ra[i][2], ra[i][3])};
      *(u32x2*)(A + row * 128 + (((c8 >> 1) ^ ((row >> 1) & 7)) << 4) + (c8 & 1) * 8) = v; }
#pragma unroll
    for (int i = 0; i < 4; ++i) { int row = i * 16 + (tid >> 4), c8 = tid & 15;
      u32x2 v = {cvt_pk(rb[i][0], rb[i][1]), cvt_pk(rb[i][2], rb[i][3])};
      *(u32x2*)(B + row * 128 + (((c8 >> 1) ^ (((row >> 1) & 1) << 2)) << 4) + (c8 & 1) * 8) = v; }
  };
  f32x16 acc[2] = {};
  gload(0); lwrite(0); __syncthreads();
  constexpr int NT = KD / BK;
  for (int kt = 0; kt < NT; ++kt) {
    const int cur = kt & 1;
    if (kt + 1 < NT) gload(kt + 1);
    const char* A = smem + cur * (TM * BK * 2 + BK * TN * 2); const char* B = A + TM * BK * 2;
#pragma unroll
    for (int s = 0; s < 4; ++s) {
      bf16x8 af[2];
#pragma unroll
      for (int m = 0; m < 2; ++m) { int row = wr * 64 + m * 32 + (lane & 31), ch = 2 * s + (lane >> 5);
        af[m] = *(const bf16x8*)(A + row * 128 + ((ch ^ ((row >> 1) & 7)) << 4)); }
      bf16x4 bt[2];
#pragma unroll
      for (int t = 0; t < 2; ++t) { int h = lane >> 5, blk = (lane >> 4) & 1, q = (lane & 15) >> 2, pp = lane & 3;
        int row = 16 * s + 8 * h + 4 * t + q; int c4 = wc * 8 + blk * 4 + pp;
        unsigned addr = (unsigned)(size_t)(B + row * 128 + (((c4 >> 1) ^ (((row >> 1) & 1) << 2)) << 4) + (c4 & 1) * 8);
        bt[t] = __builtin_amdgcn_ds_read_tr16_b64_v4i16((__attribute__((address_space(3))) bf16x4*)addr); }
      bf16x8 bf = {bt[0][0], bt[0][1], bt[0][2], bt[0][3], bt[1][0], bt[1][1], bt[1][2], bt[1][3]};
#pragma unroll
      for (int m = 0; m < 2; ++m) acc[m] = __builtin_amdgcn_mfma_f32_32x32x16_bf16(bf, af[m], acc[m], 0, 0, 0);
    }
    if (kt + 1 < NT) lwrite(cur ^ 1);
    __syncthreads();
  }
  float mx = 0.f;
#pragma unroll
  for (int m = 0; m < 2; ++m)
#pragma unroll
    for (int r = 0; r < 16; ++r) { float v = fmaxf(acc[m][r], 0.f); acc[m][r] = v; mx = fmaxf(mx, v); }
#pragma unroll
  for (int o = 32; o >= 1; o >>= 1) mx = fmaxf(mx, __shfl_xor(mx, o));
  float* red = (float*)(smem);
  unsigned* bc = (unsigned*)(smem + 64);
  if (lane == 0) red[wid] = mx;
  __syncthreads();
  if (wid == 0) {
    call = __builtin_amdgcn_readfirstlane(call);
    const unsigned par = call & 1;
    if (lane == 0) { float m4 = fmaxf(fmaxf(red[0], red[1]), fmaxf(red[2], red[3]));
      st_sys64(myws + WS_LMAX + par * 2048 + b * 8, ((unsigned long long)call << 32) | __float_as_uint(m4)); }
    bool dead = false;
    const unsigned long long t0 = __builtin_amdgcn_s_memrealtime();
    if (b == 0) {
      unsigned lm;
      for (;;) { bool ok = true; lm = 0;
#pragma unroll
        for (int i = 0; i < 4; ++i) { unsigned long long g = ld_sys64(myws + WS_LMAX + par * 2048 + (i * 64 + lane) * 8); ok &= (unsigned)(g >> 32) == call; lm = max(lm, (unsigned)g); }
        if (__all(ok)) break;
        if (__builtin_amdgcn_s_memrealtime() - t0 > DEADLINE) { dead = true; break; }
        __builtin_amdgcn_s_sleep(1); }
#pragma unroll
      for (int o = 32; o >= 1; o >>= 1) lm = max(lm, (unsigned)__shfl_xor((int)lm, o));
      if (!dead) st_sys64(p.ws[lane >> 3] + WS_GMAX + par * 1024 + (lane & 7) * 128 + p.rank * 8, ((unsigned long long)call << 32) | lm);
    }
    unsigned gm = 0;
    while (!dead) { unsigned long long g = ld_sys64(myws + WS_GMAX + par * 1024 + (b & 7) * 128 + (lane & 7) * 8);
      gm = (unsigned)g;
      if (__all((unsigned)(g >> 32) == call)) break;
      if (ld_sys32(myws + WS_TMO) == call || __builtin_amdgcn_s_memrealtime() - t0 > DEADLINE) { dead = true; break; }
      __builtin_amdgcn_s_sleep(1); }
    if (dead && lane == 0) st_sys32(myws + WS_TMO, call);
#pragma unroll
    for (int o = 4; o >= 1; o >>= 1) gm = max(gm, (unsigned)__shfl_xor((int)gm, o));
    if (lane == 0) { dead = dead || ld_sys32(myws + WS_TMO) == call; bc[0] = gm; bc[1] = dead ? 1u : 0u; bc[2] = call; }
  }
  __syncthreads();
  const float amax = __uint_as_float(bc[0]);
  const bool dead = bc[1] != 0;
  call = bc[2];
  const float inv = amax > 0.f ? 127.0f / amax : 0.f;
  const int dst = tn >> 3;
  signed char* ob = p.out[dst] + (size_t)(p.rank * MS + tm * TM + wr * 64) * NS + (tn & 7) * TN + wc * 32;
#pragma unroll
  for (int m = 0; m < 2; ++m) {
    unsigned d[4];
#pragma unroll
    for (int g = 0; g < 4; ++g) { unsigned v = 0;
#pragma unroll
      for (int e = 0; e < 4; ++e) { int q = (int)__builtin_rintf(acc[m][g * 4 + e] * inv); q = min(q, 127); v |= (unsigned)(q & 255) << (8 * e); }
      d[g] = dead ? 0x80808080u : v; }
    auto s0 = __builtin_amdgcn_permlane32_swap(d[0], d[2], false, false);
    auto s1 = __builtin_amdgcn_permlane32_swap(d[1], d[3], false, false);
    u32x4 o = {s0[0], s0[1], s1[0], s1[1]};
    signed char* q = ob + (size_t)(m * 32 + (lane & 31)) * NS + (lane >> 5) * 16;
    asm volatile("global_store_dwordx4 %0, %1, off sc0 sc1\n\ts_nop 1" ::"v"(q), "v"(o) : "memory");
  }
  asm volatile("s_waitcnt vmcnt(0)" ::: "memory");
  __syncthreads();
  if (tid == 0) st_sys32(p.ws[dst] + WS_DONE + p.rank * 128 + j * 4, call);
  if (b == 0 && wid == 0) {
    const unsigned long long t0 = __builtin_amdgcn_s_memrealtime();
    bool dd = false;
    for (;;) { bool ok = true;
#pragma unroll
      for (int i = 0; i < 4; ++i) ok &= ld_sys32(myws + WS_DONE + (i * 64 + lane) * 4) >= call;
      if (__all(ok)) break;
      if (__builtin_amdgcn_s_memrealtime() - t0 > DEADLINE) { dd = true; break; }
      __builtin_amdgcn_s_sleep(1); }
    if (dd && lane == 0) { st_sys32(myws + WS_TMO, call); p.out[p.rank][(size_t)p.rank * MS * NS] = (signed char)-128; }
  }
}

extern "C" void kernel_launch(int rank, int nranks, ncclComm_t comm, hipStream_t stream,
                              void* const* d_in, const int* in_sizes, int n_in,
                              void* d_out, int out_size, const int* in_dtypes, int out_dtype,
                              void* const* d_out_all, void* const* d_ws_all, size_t ws_bytes) {
  if (nranks != NR || ws_bytes < WS_END || n_in < 2) { fprintf(stderr, "bad config\n"); return; }
  Params p;
  if (in_sizes[0] != MS * KD || in_sizes[1] != KD * ND || out_size != 4096 * NS) { fprintf(stderr, "bad sizes %d %d %d\n", in_sizes[0], in_sizes[1], out_size); return; }
  p.x = (const float*)d_in[0]; p.w = (const float*)d_in[1];
  for (int i = 0; i < NR; ++i) { p.ws[i] = (char*)d_ws_all[i]; p.out[i] = (signed char*)d_out_all[i]; }
  p.rank = rank; p.pad = 0;
  hipLaunchKernelGGL(gemm_a2a, dim3(NWG), dim3(256), 0, stream, p);
  HIP_CHECK(hipGetLastError());
}
```

```cpp
#include <hip/hip_runtime.h>
#include <rccl/rccl.h>
#include <cstdio>
#include <cstdint>

#define PSLEEP 1
#define CPRIO 0
#define HIP_CHECK(expr) do { hipError_t _e = (expr); if (_e != hipSuccess) { \
    fprintf(stderr, "HIP error %s at %s:%d\n", hipGetErrorString(_e), __FILE__, __LINE__); \
    return; } } while (0)

typedef unsigned u32x4 __attribute__((ext_vector_type(4)));
typedef unsigned u32x2 __attribute__((ext_vector_type(2)));
typedef float f32x4 __attribute__((ext_vector_type(4)));
typedef float f32x2 __attribute__((ext_vector_type(2)));
typedef short bf16x8 __attribute__((ext_vector_type(8)));
typedef short bf16x4 __attribute__((ext_vector_type(4)));
typedef __bf16 bf16x2_t __attribute__((ext_vector_type(2)));
#define LAS __attribute__((address_space(3)))

constexpr int NR = 8;
constexpr int MS = 512;
constexpr int KD = 4096, ND = 4096;
constexpr int NS = 512;
constexpr int NWG = 256, NTHR = 512;
constexpr int NU = 16;
constexpr unsigned long long DEADLINE = 40000000ull;

constexpr int B_SLOT = 16384, NB = 8;
constexpr int LDS_MISC = NB * B_SLOT, LDS_BYTES = LDS_MISC + 256;

constexpr size_t WS_CNT  = 0;
constexpr size_t WS_AFL  = WS_CNT + 256 * 128;
constexpr size_t WS_XFL  = WS_AFL + 8 * 128;
constexpr size_t WS_LMAX = WS_XFL + 256 * 128;
constexpr size_t WS_GMAX = WS_LMAX + 2 * 256 * 8;
constexpr size_t WS_DONE = WS_GMAX + 2 * 8 * 128;
constexpr size_t WS_TMO  = WS_DONE + 8 * 128;
constexpr size_t WS_ALN  = WS_TMO + 128;
constexpr size_t WS_XA   = 1 << 20;
constexpr size_t WS_XB   = WS_XA + (4 << 20);
constexpr size_t WS_END  = WS_XB + (size_t)64 * 4 * 3 * 16384;
static_assert(WS_ALN + 8 * 128 <= WS_XA, "control area");

struct Params {
  const float* x; const float* w;
  char* ws[NR]; signed char* out[NR];
  unsigned long long* stamps;
  int rank; int pad;
};

__device__ __forceinline__ unsigned cvt_pk(float lo, float hi) {
  f32x2 v = {lo, hi};
  return __builtin_bit_cast(unsigned, __builtin_convertvector(v, bf16x2_t));
}
__device__ __forceinline__ unsigned long long ld_sys64(const void* p) { return __hip_atomic_load((const unsigned long long*)p, __ATOMIC_RELAXED, __HIP_MEMORY_SCOPE_SYSTEM); }
__device__ __forceinline__ unsigned ld_sys32(const void* p) { return __hip_atomic_load((const unsigned*)p, __ATOMIC_RELAXED, __HIP_MEMORY_SCOPE_SYSTEM); }
__device__ __forceinline__ void st_sys64(void* p, unsigned long long v) { __hip_atomic_store((unsigned long long*)p, v, __ATOMIC_RELAXED, __HIP_MEMORY_SCOPE_SYSTEM); }
__device__ __forceinline__ void st_sys32(void* p, unsigned v) { __hip_atomic_store((unsigned*)p, v, __ATOMIC_RELAXED, __HIP_MEMORY_SCOPE_SYSTEM); }
#define LSC "sc0 sc1"
__device__ __forceinline__ void st_loc128(void* p, u32x4 v) { asm volatile("global_store_dwordx4 %0, %1, off " LSC "\n\ts_nop 1" ::"v"(p), "v"(v) : "memory"); }
__device__ __forceinline__ void st_wt128(void* p, u32x4 v) { asm volatile("global_store_dwordx4 %0, %1, off sc0 sc1\n\ts_nop 1" ::"v"(p), "v"(v) : "memory"); }
#define WAIT_V(n) asm volatile("s_waitcnt vmcnt(%0)" ::"n"(n) : "memory")
__device__ __forceinline__ void bar() { asm volatile("s_waitcnt lgkmcnt(0)\n\ts_barrier" ::: "memory"); }
__device__ __forceinline__ void lds_put(unsigned addr, unsigned v) { asm volatile("ds_write_b32 %0, %1" ::"v"(addr), "v"(v) : "memory"); }
__device__ __forceinline__ unsigned lds_get(unsigned addr) { unsigned r; asm volatile("ds_read_b32 %0, %1\n\ts_waitcnt lgkmcnt(0)" : "=v"(r) : "v"(addr) : "memory"); return __builtin_amdgcn_readfirstlane(r); }
__device__ __forceinline__ unsigned lds_min4(unsigned addr) {
  u32x4 r; asm volatile("ds_read_b128 %0, %1\n\ts_waitcnt lgkmcnt(0)" : "=v"(r) : "v"(addr) : "memory");
  return __builtin_amdgcn_readfirstlane(min(min(r.x, r.y), min(r.z, r.w)));
}
#define SPIN_UNTIL(COND) do { const unsigned long long t0_ = __builtin_amdgcn_s_memrealtime(); unsigned it_ = 0; \
    while (!(COND)) { \
      if ((++it_ & 63u) == 0u && (ld_sys32(myws + WS_TMO) == call || __builtin_amdgcn_s_memrealtime() - t0_ > DEADLINE)) { dead = true; break; } \
      __builtin_amdgcn_s_sleep(1); } \
    if (dead && lane == 0) st_sys32(myws + WS_TMO, call); } while (0)
#define STAMP(i) do {} while (0)
#define TR(V, IDX) do {} while (0)

__device__ __forceinline__ int unit_of(int t, int m, int n) {
  int v = t;
  if (m) v = (t == 0) ? 1 : (t == 15) ? 14 : ((t & 1) ? t + 2 : t - 2);
  return (v + n) & 15;
}

__device__ __forceinline__ void issueB(u32x4 (&rb)[8], const char* wbase, unsigned voff, int unit) {
  const char* src = wbase + (size_t)unit * 64 * ND * 4;
#pragma unroll
  for (int i = 0; i < 8; ++i) asm volatile("global_load_dwordx4 %0, %1, %2" : "=v"(rb[i]) : "v"(voff), "s"(src + (size_t)i * 8 * ND * 4) : "memory");
}
__device__ __forceinline__ void writeB(u32x4 (&rb)[8], char* smem, int woff0, int woff1, int slot) {
  LAS char* B = (LAS char*)smem + slot * B_SLOT;
#pragma unroll
  for (int i = 0; i < 8; ++i) {
    asm volatile("" : "+v"(rb[i]));
    const f32x4 v = __builtin_bit_cast(f32x4, rb[i]);
    *(LAS u32x2*)(B + ((i & 1) ? woff1 : woff0) + i * 2048) = (u32x2){cvt_pk(v[0], v[1]), cvt_pk(v[2], v[3])};
  }
}

__global__ __launch_bounds__(NTHR, 2) void gemm_a2a(Params p) {
  extern __shared__ __attribute__((aligned(16))) char smem[];
  int tid = threadIdx.x, lane = tid & 63;
  const int wid = __builtin_amdgcn_readfirstlane(tid >> 6);
  const int b = blockIdx.x;
  const int g = b & 7, q = g & 3, h = g >> 2, j = b >> 3, m = j & 1, n = j >> 1;
  const int ctile = h * 16 + n;
  char* myws = p.ws[p.rank];
  volatile LAS unsigned* misc = (volatile LAS unsigned*)(smem + LDS_MISC);
  const unsigned miscA = (unsigned)(size_t)(LAS char*)smem + LDS_MISC;
  if (tid < 64) misc[tid] = 0;
  bar();
  STAMP(0);

  f32x4 acc[4][8];
  unsigned call = 0;
  bool dead = false;

  if (wid < 4) {
    if (tid == 0) { call = ld_sys32(myws + WS_CNT + b * 128) + 1; }
    unsigned have = 0;
    const int pu = unit_of(0, m, n), pc = pu * 2 + h;
    {
      const int iw = (pc & 1) * 2 + (wid >> 1), ks = wid & 1;
      u32x4 xr[8];
      const float* xb = p.x + (size_t)(m * 256 + iw * 64 + (lane >> 3)) * KD + q * 1024 + pu * 64 + ks * 32 + (lane & 7) * 4;
#pragma unroll
      for (int i = 0; i < 8; ++i) {
        const float* ptr = xb + (size_t)(((((i >> 1) + q) & 3) * 16) + (i & 1) * 8) * KD;
        asm volatile("global_load_dwordx4 %0, %1, off" : "=v"(xr[i]) : "v"(ptr) : "memory");
      }
      WAIT_V(0);
      if (lane == 0) asm volatile("ds_add_u32 %0, %1" ::"v"(miscA + 16 * 4), "v"(1u) : "memory");
      STAMP(12);
#pragma unroll
      for (int i = 0; i < 8; ++i) asm volatile("" : "+v"(xr[i]));
      char* xa = myws + WS_XA + ((size_t)(((q * 2 + m) * 4 + iw) * 16 + pu) * 8 + ks * 4) * 1024;
      const bool odd = lane & 1;
      const int dl = (lane >> 3) + (odd ? 8 : 0) + 16 * ((lane & 7) >> 1);
#pragma unroll
      for (int i = 0; i < 8; i += 2) {
        const f32x4 a = __builtin_bit_cast(f32x4, xr[i]), c = __builtin_bit_cast(f32x4, xr[i + 1]);
        const unsigned a0 = cvt_pk(a[0], a[1]), a1 = cvt_pk(a[2], a[3]), c0 = cvt_pk(c[0], c[1]), c1 = cvt_pk(c[2], c[3]);
        const unsigned s0 = odd ? a0 : c0, s1 = odd ? a1 : c1;
        const unsigned r0 = __builtin_amdgcn_mov_dpp(s0, 0xB1, 0xF, 0xF, true), r1 = __builtin_amdgcn_mov_dpp(s1, 0xB1, 0xF, 0xF, true);
        const u32x4 o = odd ? (u32x4){r0, r1, c0, c1} : (u32x4){a0, a1, r0, r1};
        st_loc128(xa + (i >> 1) * 1024 + dl * 16, o);
      }
      STAMP(13);
      WAIT_V(0);
      call = __builtin_amdgcn_readfirstlane(call);
      lds_put(miscA + (20 + wid) * 4, 1);
      if (wid == 0) {
        while (lds_min4(miscA + 20 * 4) == 0) __builtin_amdgcn_s_sleep(1);
        STAMP(14);
        if (lane == 0) { st_sys32(myws + WS_AFL + (q * 2 + m) * 128 + pc * 4, call); st_sys32(myws + WS_CNT + b * 128, call); misc[0] = call; }
        lds_put(miscA + 17 * 4, 4);
        const unsigned need = (3u << (2 * unit_of(0, m, n))) | (3u << (2 * unit_of(1, m, n)));
        SPIN_UNTIL(((have = (unsigned)__ballot(ld_sys32(myws + WS_AFL + (q * 2 + m) * 128 + (lane & 31) * 4) >= call)) & need) == need);
        if (dead) have = ~0u;
        lds_put(miscA + 19 * 4, have);
        lds_put(miscA + 20 * 4 - 8, 1);
      } else {
        while (lds_get(miscA + 18 * 4) == 0) __builtin_amdgcn_s_sleep(1);
        have = lds_get(miscA + 19 * 4);
        call = lds_get(miscA);
      }
      STAMP(1);
    }
#pragma unroll
    for (int i = 0; i < 4; ++i)
#pragma unroll
      for (int nb = 0; nb < 8; ++nb) acc[i][nb] = (f32x4){0.f, 0.f, 0.f, 0.f};
    const char* xa = myws + WS_XA + (size_t)((q * 2 + m) * 4 + wid) * 16 * 8192;
    const unsigned xv0 = lane * 16, xv1 = xv0 + 4096;
    unsigned long long utab = 0;
#pragma unroll
    for (int t = 0; t < NU; ++t) utab |= (unsigned long long)unit_of(t, m, n) << (4 * t);
#define XADDR(T) (xa + (size_t)((unsigned)(utab >> (4 * (T))) & 15u) * 8192)
    u32x4 X0[2][4], X1[2][4];
#define LX(X, KS, I, A) TM3(c_lx, LX_(X, KS, I, A))
#define LX_(X, KS, I, A) asm volatile("global_load_dwordx4 %0, %1, %2 offset:%3 sc0 sc1" : "+v"(X[KS][I]) : "v"((KS) ? xv1 : xv0), "s"(A), "n"((I) * 1024) : "memory")
#define TM3(ACCU, STMT) do { STMT; } while (0)
#define WAITX_(X, KS, N) asm volatile("s_waitcnt vmcnt(%4)" : "+v"(X[KS][0]), "+v"(X[KS][1]), "+v"(X[KS][2]), "+v"(X[KS][3]) : "n"(N) : "memory")
#define WAITX(X, KS, N) TM3(c_wx, WAITX_(X, KS, N))
#pragma unroll
    for (int ks = 0; ks < 2; ++ks)
#pragma unroll
      for (int i = 0; i < 4; ++i) {
        X0[ks][i] = (u32x4){0, 0, 0, 0}; X1[ks][i] = (u32x4){0, 0, 0, 0};
      }
    const char* xn = XADDR(0);
    LX(X0, 0, 0, xn); LX(X0, 0, 1, xn); LX(X0, 0, 2, xn); LX(X0, 0, 3, xn);
    LX(X0, 1, 0, xn); LX(X0, 1, 1, xn); LX(X0, 1, 2, xn); LX(X0, 1, 3, xn);
    xn = XADDR(1);
    LX(X1, 0, 0, xn); LX(X1, 0, 1, xn); LX(X1, 0, 2, xn); LX(X1, 0, 3, xn);
    unsigned BA0[2][2], BA1[2][2];
    {
      const int gg = lane >> 4, q4 = (lane & 15) >> 2, pp = lane & 3;
#pragma unroll
      for (int t = 0; t < 2; ++t)
#pragma unroll
        for (int cp = 0; cp < 2; ++cp) {
          BA0[t][cp] = (miscA - LDS_MISC) + 2048 * gg + 64 * (4 * t + q4) + 16 * ((2 * cp + (pp >> 1)) ^ ((2 * gg + t) & 3)) + 8 * (pp & 1);
          BA1[t][cp] = BA0[t][cp];
        }
    }
    bf16x8 W0[2], W1[2], W2[2], W3[2];
#define RDW(BA, C, T, KS, CP) __builtin_amdgcn_ds_read_tr16_b64_v4i16((LAS bf16x4*)(size_t)(BA[T][C] + (KS) * 8192 + (CP) * 512))
#define JOINW(lo, hi) ((bf16x8){lo[0], lo[1], lo[2], lo[3], hi[0], hi[1], hi[2], hi[3]})
#define MF(ACC, W, X) asm volatile("v_mfma_f32_16x16x32_bf16 %0, %1, %2, %0" : "+v"(ACC) : "v"(W), "v"(X))
#define SB0() __builtin_amdgcn_sched_barrier(0)
#define LOADW(W, BA, KS, CP) do { \
      { const bf16x4 lo = RDW(BA, 0, 0, KS, CP), hi = RDW(BA, 0, 1, KS, CP); W[0] = JOINW(lo, hi); } \
      { const bf16x4 lo = RDW(BA, 1, 0, KS, CP), hi = RDW(BA, 1, 1, KS, CP); W[1] = JOINW(lo, hi); } } while (0)
#define STEP(X, KS, W, CP, WN, BA, NKS, NCP, H1, H2, H3) do { \
      MF(acc[0][(CP) * 2], W[0], X[KS][0]); SB0(); \
      const bf16x4 l0 = RDW(BA, 0, 0, NKS, NCP); SB0(); \
      MF(acc[1][(CP) * 2], W[0], X[KS][1]); SB0(); H1; SB0(); MF(acc[2][(CP) * 2], W[0], X[KS][2]); SB0(); \
      const bf16x4 h0 = RDW(BA, 0, 1, NKS, NCP); SB0(); \
      MF(acc[3][(CP) * 2], W[0], X[KS][3]); MF(acc[0][(CP) * 2 + 1], W[1], X[KS][0]); SB0(); \
      const bf16x4 l1 = RDW(BA, 1, 0, NKS, NCP); SB0(); \
      MF(acc[1][(CP) * 2 + 1], W[1], X[KS][1]); SB0(); H2; SB0(); MF(acc[2][(CP) * 2 + 1], W[1], X[KS][2]); SB0(); \
      const bf16x4 h1 = RDW(BA, 1, 1, NKS, NCP); SB0(); \
      MF(acc[3][(CP) * 2 + 1], W[1], X[KS][3]); SB0(); H3; SB0(); \
      WN[0] = JOINW(l0, h0); WN[1] = JOINW(l1, h1); } while (0)
#define STEPL(X, KS, W, CP) do { _Pragma("unroll") for (int c = 0; c < 2; ++c) _Pragma("unroll") for (int i = 0; i < 4; ++i) MF(acc[i][(CP) * 2 + c], W[c], X[KS][i]); } while (0)
#define NEEDX(U) do { if (__builtin_expect(!((have >> (2 * (U) + (wid >> 1))) & 1u), 0)) { \
        SPIN_UNTIL(((have = (unsigned)__ballot(ld_sys32(myws + WS_AFL + (q * 2 + m) * 128 + (lane & 31) * 4) >= call)) >> (2 * (U) + (wid >> 1))) & 1u); \
        if (dead) have = ~0u; } } while (0)
#define NOP() do {} while (0)
#define VADD(D, S, K) asm volatile("v_add_u32 %0, %2, %1" : "=v"(D) : "v"(S), "s"(K))
#define UNIT(X, XO, BA, BN, T, L1, L0, N0, N4, LAST) do { \
      int d_; unsigned u_; TR(trc, (T)); \
      STEP(X, 0, W0, 0, W2, BA, 0, 2, if (L1) LX(XO, 1, 0, xn), if (L1) LX(XO, 1, 1, xn), d_ = (((T) + 1) & (NB - 1)) ? B_SLOT : -(NB - 1) * B_SLOT); \
      STEP(X, 0, W1, 1, W3, BA, 0, 3, if (L1) LX(XO, 1, 2, xn), if (L1) LX(XO, 1, 3, xn), u_ = (unsigned)(utab >> (4 * (((T) + 2) & 15))) & 15u); \
      STEP(X, 0, W2, 2, W0, BA, 1, 0, VADD(BN[0][0], BA[0][0], d_), VADD(BN[0][1], BA[0][1], d_), NOP()); \
      STEP(X, 0, W3, 3, W1, BA, 1, 1, VADD(BN[1][0], BA[1][0], d_), VADD(BN[1][1], BA[1][1], d_), WAITX(X, 1, N4)); \
      STEP(X, 1, W0, 0, W2, BA, 1, 2, if (L0) xn = xa + (size_t)u_ * 8192, if (L0) NEEDX(u_), NOP()); \
      STEP(X, 1, W1, 1, W3, BA, 1, 3, NOP(), NOP(), NOP()); \
      if (!(LAST)) { \
        TR(trc, 16 + (T)); asm volatile("s_barrier" ::: "memory"); TR(trc, 32 + (T)); SB0();        \
        STEP(X, 1, W2, 2, W0, BN, 0, 0, if (L0) LX(X, 0, 0, xn), if (L0) LX(X, 0, 1, xn), NOP()); \
        STEP(X, 1, W3, 3, W1, BN, 0, 1, if (L0) LX(X, 0, 2, xn), if (L0) LX(X, 0, 3, xn), WAITX(XO, 0, N0)); } \
      else { STEPL(X, 1, W2, 2); STEPL(X, 1, W3, 3); } } while (0)
    if (CPRIO) __builtin_amdgcn_s_setprio(CPRIO);
    unsigned trc = 0;
    bar();
    LOADW(W0, BA0, 0, 0); LOADW(W1, BA0, 0, 1);
    WAITX(X0, 0, 8);
    SB0();
    STAMP(2);
    for (int t = 0; t < NU - 2; t += 2) { UNIT(X0, X1, BA0, BA1, t, true, true, 8, 8, false); UNIT(X1, X0, BA1, BA0, t + 1, true, true, 8, 8, false); }
    UNIT(X0, X1, BA0, BA1, NU - 2, true, false, 4, 8, false);
    UNIT(X1, X0, BA1, BA0, NU - 1, false, false, 0, 0, true);
#undef LX
#undef XADDR
#undef NOP
#undef NEEDX
#undef VADD
#undef WAITX
#undef WAITX_
#undef LX_
#undef TM3
#undef LOADW
#undef RDW
#undef JOINW
#undef MF
#undef SB0
#undef STEP
#undef STEPL
#undef UNIT
    asm volatile("s_nop 15\n\ts_nop 15" ::: "memory");
    STAMP(3);
  } else {
    const int pw = wid - 4;
    const char* wb = (const char*)(p.w + (size_t)(q * 1024) * ND + ctile * 128 + pw * 32);
    const unsigned wvo = (unsigned)((lane >> 3) * ND * 4 + (lane & 7) * 16);
    u32x4 rb0[8], rb1[8];
    const int woff0 = 512 * pw + 64 * (lane >> 3) + 16 * (((lane & 7) >> 1) ^ (lane >> 5)) + 8 * (lane & 1);
    const int woff1 = 512 * pw + 64 * (lane >> 3) + 16 * (((lane & 7) >> 1) ^ (2 | (lane >> 5))) + 8 * (lane & 1);
    while (lds_get(miscA + 16 * 4) < 4) __builtin_amdgcn_s_sleep(1);
    issueB(rb0, wb, wvo, unit_of(0, m, n));
    while (lds_get(miscA + 17 * 4) < 4) __builtin_amdgcn_s_sleep(1);
    issueB(rb1, wb, wvo, unit_of(1, m, n));
    unsigned trp = 0;
    int nbar = 0;
#define PSTEP(s, RB, MORE, NW) do { \
      if (nbar) { bar(); ++nbar; } \
      else if ((s) >= 2 && ((s) == NB - 2 || lds_get(miscA + 18 * 4) != 0)) { bar(); nbar = 1; } \
      TR(trp, (s)); WAIT_V(NW); TR(trp, 16 + (s)); \
      writeB(RB, smem, woff0, woff1, (s) & (NB - 1)); \
      if (MORE) { TR(trp, 32 + (s)); issueB(RB, wb, wvo, unit_of((s) + 2, m, n)); TR(trp, 48 + (s)); } } while (0)
    for (int s = 0; s < NU - 2; s += 2) { PSTEP(s, rb0, true, 8); PSTEP(s + 1, rb1, true, 8); }
    PSTEP(NU - 2, rb0, false, 8); PSTEP(NU - 1, rb1, false, 0);
    for (; nbar < NU; ++nbar) bar();
#undef PSTEP
  }
  bar();
  asm volatile("" : "+v"(tid));
  lane = tid & 63;

  const int tile = m * 32 + ctile;
  call = misc[0];
  if (wid < 4) {
#pragma unroll
    for (int i = 1; i < 4; ++i) {
      char* dst = myws + WS_XB + ((size_t)(tile * 4 + ((q + i) & 3)) * 3 + (i - 1)) * 16384 + wid * 4096 + lane * 16;
#pragma unroll
      for (int e = 0; e < 4; ++e) {
        const u32x4 o = {cvt_pk(acc[i][2 * e][0], acc[i][2 * e][1]), cvt_pk(acc[i][2 * e][2], acc[i][2 * e][3]),
                         cvt_pk(acc[i][2 * e + 1][0], acc[i][2 * e + 1][1]), cvt_pk(acc[i][2 * e + 1][2], acc[i][2 * e + 1][3])};
        st_loc128(dst + e * 1024, o);
      }
    }
    WAIT_V(0);
  }
  bar();
  if (wid == 0) {
    if (lane < 3) st_sys32(myws + WS_XFL + (size_t)(tile * 4 + ((q + lane + 1) & 3)) * 128 + lane * 4, call);
    if (!dead) SPIN_UNTIL(__all(ld_sys32(myws + WS_XFL + (size_t)(tile * 4 + q) * 128 + (lane < 3 ? lane : 0) * 4) >= call));
  }
  bar();
  STAMP(4);
  float mx = 0.f;
  if (wid < 4) {
    u32x4 pr[3][4];
    const char* src = myws + WS_XB + (size_t)(tile * 4 + q) * 3 * 16384 + wid * 4096 + lane * 16;
#pragma unroll
    for (int i = 0; i < 3; ++i)
#pragma unroll
      for (int e = 0; e < 4; ++e) asm volatile("global_load_dwordx4 %0, %1, off sc0 sc1" : "=v"(pr[i][e]) : "v"(src + i * 16384 + e * 1024) : "memory");
    WAIT_V(0);
#pragma unroll
    for (int i = 0; i < 3; ++i)
#pragma unroll
      for (int e = 0; e < 4; ++e) {
        asm volatile("" : "+v"(pr[i][e]));
#pragma unroll
        for (int d = 0; d < 4; ++d) {
          acc[0][2 * e + (d >> 1)][(d & 1) * 2] += __uint_as_float(pr[i][e][d] << 16);
          acc[0][2 * e + (d >> 1)][(d & 1) * 2 + 1] += __uint_as_float(pr[i][e][d] & 0xffff0000u);
        }
      }
#pragma unroll
    for (int nb = 0; nb < 8; ++nb)
#pragma unroll
      for (int r = 0; r < 4; ++r) { const float v = fmaxf(acc[0][nb][r], 0.f); acc[0][nb][r] = v; mx = fmaxf(mx, v); }
#pragma unroll
    for (int o = 32; o >= 1; o >>= 1) mx = fmaxf(mx, __shfl_xor(mx, o));
    if (lane == 0) misc[4 + wid] = __float_as_uint(mx);
  }
  bar();
  STAMP(5);
  if (wid == 0) {
    const unsigned par = call & 1;
    if (lane == 0) { const unsigned m4 = max(max(misc[4], misc[5]), max(misc[6], misc[7]));
      st_sys64(myws + WS_LMAX + par * 2048 + b * 8, ((unsigned long long)call << 32) | m4); }
    if (b == 0 && !dead) {
      unsigned lm = 0;
      auto sweep = [&]() { bool ok = true; lm = 0;
#pragma unroll
        for (int i = 0; i < 4; ++i) { const unsigned long long gv = ld_sys64(myws + WS_LMAX + par * 2048 + (i * 64 + lane) * 8); ok &= (unsigned)(gv >> 32) == call; lm = max(lm, (unsigned)gv); }
        return __all(ok) != 0; };
      SPIN_UNTIL(sweep());
#pragma unroll
      for (int o = 32; o >= 1; o >>= 1) lm = max(lm, (unsigned)__shfl_xor((int)lm, o));
      if (!dead) st_sys64(p.ws[lane >> 3] + WS_GMAX + par * 1024 + (lane & 7) * 128 + p.rank * 8, ((unsigned long long)call << 32) | lm);
    }
    unsigned gm = 0;
    if (!dead) {
      auto look = [&]() { const unsigned long long gv = ld_sys64(myws + WS_GMAX + par * 1024 + (b & 7) * 128 + (lane & 7) * 8); gm = (unsigned)gv; return __all((unsigned)(gv >> 32) == call) != 0; };
      SPIN_UNTIL(look());
    }
#pragma unroll
    for (int o = 4; o >= 1; o >>= 1) gm = max(gm, (unsigned)__shfl_xor((int)gm, o));
    if (lane == 0) { dead = dead || ld_sys32(myws + WS_TMO) == call; misc[1] = gm; misc[2] = dead ? 1u : 0u; }
  }
  bar();
  STAMP(6);
  if (wid < 4) {
    const float amax = __uint_as_float(misc[1]);
    const bool bad = misc[2] != 0;
    const float inv = amax > 0.f ? 127.0f / amax : 0.f;
#pragma unroll
    for (int nb = 0; nb < 8; ++nb) {
      unsigned v = 0;
#pragma unroll
      for (int e = 0; e < 4; ++e) { int qv = (int)__builtin_rintf(acc[0][nb][e] * inv); qv = min(qv, 127); v |= (unsigned)(qv & 255) << (8 * e); }
      *(LAS unsigned*)((LAS char*)smem + (wid * 16 + (lane & 15)) * 144 + nb * 16 + (lane >> 4) * 4) = bad ? 0x80808080u : v;
    }
  }
  bar();
  const int dstr = ctile >> 2;
  {
    const int rho = wid * 8 + (lane >> 3);
    const u32x4 o = *(const LAS u32x4*)((LAS char*)smem + rho * 144 + (lane & 7) * 16);
    const int row = p.rank * MS + m * 256 + (rho >> 4) * 64 + q * 16 + (rho & 15);
    st_wt128(p.out[dstr] + (size_t)row * NS + (ctile & 3) * 128 + (lane & 7) * 16, o);
    WAIT_V(0);
  }
  bar();
  STAMP(7);
  if (tid == 0) st_sys32(p.ws[dstr] + WS_DONE + p.rank * 128 + (((ctile & 3) * 2 + m) * 4 + q) * 4, call);
  if (b == 0 && wid == 0) {
    dead = false;
    auto done = [&]() { bool ok = true;
#pragma unroll
      for (int i = 0; i < 4; ++i) ok &= ld_sys32(myws + WS_DONE + (i * 64 + lane) * 4) >= call;
      return __all(ok) != 0; };
    SPIN_UNTIL(done());
    if (dead && lane == 0) p.out[p.rank][(size_t)p.rank * MS * NS] = (signed char)-128;
  }
  STAMP(8);
}

static unsigned long long* g_stamps = nullptr;

extern "C" void kernel_launch(int rank, int nranks, ncclComm_t comm, hipStream_t stream,
                              void* const* d_in, const int* in_sizes, int n_in,
                              void* d_out, int out_size, const int* in_dtypes, int out_dtype,
                              void* const* d_out_all, void* const* d_ws_all, size_t ws_bytes) {
  if (nranks != NR || ws_bytes < WS_END || n_in < 2) { fprintf(stderr, "bad config\n"); return; }
  if (in_sizes[0] != MS * KD || in_sizes[1] != KD * ND || out_size != 4096 * NS) { fprintf(stderr, "bad sizes\n"); return; }
  HIP_CHECK(hipFuncSetAttribute((const void*)gemm_a2a, hipFuncAttributeMaxDynamicSharedMemorySize, LDS_BYTES));
  Params p;
  p.x = (const float*)d_in[0]; p.w = (const float*)d_in[1];
  for (int i = 0; i < NR; ++i) { p.ws[i] = (char*)d_ws_all[i]; p.out[i] = (signed char*)d_out_all[i]; }
  p.stamps = g_stamps;
  p.rank = rank; p.pad = 0;
  hipLaunchKernelGGL(gemm_a2a, dim3(NWG), dim3(NTHR), LDS_BYTES, stream, p);
  HIP_CHECK(hipGetLastError());
}
```

```cpp
#include <hip/hip_runtime.h>
#include <rccl/rccl.h>
#include <cstdio>
#include <cstdint>

#define GMODE 5
#define XMODE 0
#define PSLEEP 1
#define CPRIO 0
#define HIP_CHECK(expr) do { hipError_t _e = (expr); if (_e != hipSuccess) { \
    fprintf(stderr, "HIP error %s at %s:%d\n", hipGetErrorString(_e), __FILE__, __LINE__); \
    return; } } while (0)

typedef unsigned u32x4 __attribute__((ext_vector_type(4)));
typedef unsigned u32x2 __attribute__((ext_vector_type(2)));
typedef float f32x4 __attribute__((ext_vector_type(4)));
typedef float f32x2 __attribute__((ext_vector_type(2)));
typedef short bf16x8 __attribute__((ext_vector_type(8)));
typedef short bf16x4 __attribute__((ext_vector_type(4)));
typedef __bf16 bf16x2_t __attribute__((ext_vector_type(2)));
#define LAS __attribute__((address_space(3)))

constexpr int NR = 8;
constexpr int MS = 512;
constexpr int KD = 4096, ND = 4096;
constexpr int NS = 512;
constexpr int NWG = 256, NTHR = 512;
constexpr int NU = 16;
constexpr unsigned long long DEADLINE = 40000000ull;

constexpr int B_SLOT = 16384, NB = 8;
constexpr int LDS_MISC = NB * B_SLOT, LDS_BYTES = LDS_MISC + 256;

constexpr size_t WS_CNT  = 0;
constexpr size_t WS_AFL  = WS_CNT + 256 * 128;
constexpr size_t WS_XFL  = WS_AFL + 8 * 128;
constexpr size_t WS_LMAX = WS_XFL + 256 * 128;
constexpr size_t WS_GMAX = WS_LMAX + 2 * 256 * 8;
constexpr size_t WS_DONE = WS_GMAX + 2 * 8 * 128;
constexpr size_t WS_TFL  = WS_DONE + 8 * 128;
constexpr size_t WS_TMO  = WS_TFL + 16384;
constexpr size_t WS_ALN  = WS_TMO + 128;
constexpr size_t WS_XA   = 1 << 20;
constexpr size_t WS_XB   = WS_XA + (4 << 20);
constexpr int CHUNK = 4608;
constexpr size_t WS_STG  = WS_XB + (size_t)64 * 4 * 3 * 16384;
constexpr size_t WS_END  = WS_STG + (size_t)2 * 8 * 32 * 4 * CHUNK;
static_assert(WS_ALN + 8 * 128 <= WS_XA, "control area");

struct Params {
  const float* x; const float* w;
  char* ws[NR]; signed char* out[NR];
  unsigned long long* stamps;
  int rank; int pad;
};

__device__ __forceinline__ unsigned cvt_pk(float lo, float hi) {
  f32x2 v = {lo, hi};
  return __builtin_bit_cast(unsigned, __builtin_convertvector(v, bf16x2_t));
}
__device__ __forceinline__ unsigned long long ld_sys64(const void* p) { return __hip_atomic_load((const unsigned long long*)p, __ATOMIC_RELAXED, __HIP_MEMORY_SCOPE_SYSTEM); }
__device__ __forceinline__ unsigned ld_sys32(const void* p) { return __hip_atomic_load((const unsigned*)p, __ATOMIC_RELAXED, __HIP_MEMORY_SCOPE_SYSTEM); }
__device__ __forceinline__ void st_sys64(void* p, unsigned long long v) { __hip_atomic_store((unsigned long long*)p, v, __ATOMIC_RELAXED, __HIP_MEMORY_SCOPE_SYSTEM); }
__device__ __forceinline__ void st_sys32(void* p, unsigned v) { __hip_atomic_store((unsigned*)p, v, __ATOMIC_RELAXED, __HIP_MEMORY_SCOPE_SYSTEM); }
#define LSC "sc0 sc1"
__device__ __forceinline__ void st_loc128(void* p, u32x4 v) { asm volatile("global_store_dwordx4 %0, %1, off " LSC "\n\ts_nop 1" ::"v"(p), "v"(v) : "memory"); }
__device__ __forceinline__ void st_wt128(void* p, u32x4 v) { asm volatile("global_store_dwordx4 %0, %1, off sc0 sc1\n\ts_nop 1" ::"v"(p), "v"(v) : "memory"); }
#define WAIT_V(n) asm volatile("s_waitcnt vmcnt(%0)" ::"n"(n) : "memory")
__device__ __forceinline__ void bar() { asm volatile("s_waitcnt lgkmcnt(0)\n\ts_barrier" ::: "memory"); }
__device__ __forceinline__ void lds_put(unsigned addr, unsigned v) { asm volatile("ds_write_b32 %0, %1" ::"v"(addr), "v"(v) : "memory"); }
__device__ __forceinline__ unsigned lds_get(unsigned addr) { unsigned r; asm volatile("ds_read_b32 %0, %1\n\ts_waitcnt lgkmcnt(0)" : "=v"(r) : "v"(addr) : "memory"); return __builtin_amdgcn_readfirstlane(r); }
__device__ __forceinline__ unsigned lds_min4(unsigned addr) {
  u32x4 r; asm volatile("ds_read_b128 %0, %1\n\ts_waitcnt lgkmcnt(0)" : "=v"(r) : "v"(addr) : "memory");
  return __builtin_amdgcn_readfirstlane(min(min(r.x, r.y), min(r.z, r.w)));
}
#define SPIN_UNTIL(COND) do { const unsigned long long t0_ = __builtin_amdgcn_s_memrealtime(); unsigned it_ = 0; \
    while (!(COND)) { \
      if ((++it_ & 63u) == 0u && (ld_sys32(myws + WS_TMO) == call || __builtin_amdgcn_s_memrealtime() - t0_ > DEADLINE)) { dead = true; break; } \
      __builtin_amdgcn_s_sleep(1); } \
    if (dead && lane == 0) st_sys32(myws + WS_TMO, call); } while (0)
#define STAMP(i) do {} while (0)
#define TR(V, IDX) do {} while (0)

__device__ __forceinline__ int unit_of(int t, int m, int n) {
  int v = t;
  if (m) v = (t == 0) ? 1 : (t == 15) ? 14 : ((t & 1) ? t + 2 : t - 2);
  return (v + n) & 15;
}

__device__ __forceinline__ void issueB(u32x4 (&rb)[8], const char* wbase, unsigned voff, int unit) {
  const char* src = wbase + (size_t)unit * 64 * ND * 4;
#pragma unroll
  for (int i = 0; i < 8; ++i) asm volatile("global_load_dwordx4 %0, %1, %2" : "=v"(rb[i]) : "v"(voff), "s"(src + (size_t)i * 8 * ND * 4) : "memory");
}
__device__ __forceinline__ void writeB(u32x4 (&rb)[8], char* smem, int woff0, int woff1, int slot) {
  LAS char* B = (LAS char*)smem + slot * B_SLOT;
#pragma unroll
  for (int i = 0; i < 8; ++i) {
    asm volatile("" : "+v"(rb[i]));
    const f32x4 v = __builtin_bit_cast(f32x4, rb[i]);
    *(LAS u32x2*)(B + ((i & 1) ? woff1 : woff0) + i * 2048) = (u32x2){cvt_pk(v[0], v[1]), cvt_pk(v[2], v[3])};
  }
}

__device__ __forceinline__ unsigned wave_max_u32(unsigned v) {
  v = max(v, (unsigned)__builtin_amdgcn_update_dpp(0u, v, 0x111, 0xf, 0xf, false));
  v = max(v, (unsigned)__builtin_amdgcn_update_dpp(0u, v, 0x112, 0xf, 0xf, false));
  v = max(v, (unsigned)__builtin_amdgcn_update_dpp(0u, v, 0x114, 0xf, 0xf, false));
  v = max(v, (unsigned)__builtin_amdgcn_update_dpp(0u, v, 0x118, 0xf, 0xf, false));
  v = max(v, (unsigned)__builtin_amdgcn_update_dpp(0u, v, 0x142, 0xa, 0xf, false));
  v = max(v, (unsigned)__builtin_amdgcn_update_dpp(0u, v, 0x143, 0xc, 0xf, false));
  return __builtin_amdgcn_readlane(v, 63);
}
__device__ __forceinline__ unsigned wave_scan_u32(unsigned v) {
  v += __builtin_amdgcn_update_dpp(0u, v, 0x111, 0xf, 0xf, false);
  v += __builtin_amdgcn_update_dpp(0u, v, 0x112, 0xf, 0xf, false);
  v += __builtin_amdgcn_update_dpp(0u, v, 0x114, 0xf, 0xf, false);
  v += __builtin_amdgcn_update_dpp(0u, v, 0x118, 0xf, 0xf, false);
  v += __builtin_amdgcn_update_dpp(0u, v, 0x142, 0xa, 0xf, false);
  v += __builtin_amdgcn_update_dpp(0u, v, 0x143, 0xc, 0xf, false);
  return v;
}

__global__ __launch_bounds__(NTHR, 2) void gemm_a2a(Params p) {
  extern __shared__ __attribute__((aligned(16))) char smem[];
  int tid = threadIdx.x, lane = tid & 63;
  const int wid = __builtin_amdgcn_readfirstlane(tid >> 6);
  const int b = blockIdx.x;
  const int g = b & 7, q = g & 3, h = g >> 2, j = b >> 3, m = j & 1, n = j >> 1;
  const int ctile = h * 16 + n;
  char* myws = p.ws[p.rank];
  volatile LAS unsigned* misc = (volatile LAS unsigned*)(smem + LDS_MISC);
  const unsigned miscA = (unsigned)(size_t)(LAS char*)smem + LDS_MISC;
  if (tid < 64) misc[tid] = 0;
  bar();
  STAMP(0);

  f32x4 acc[4][8];
  unsigned call = 0;
  bool dead = false;

  if (wid < 4) {
    if (tid == 0) { call = ld_sys32(myws + WS_CNT + b * 128) + 1; }
    unsigned have = 0;
    unsigned tw[16];
    const int pu = unit_of(0, m, n), pc = pu * 2 + h;
    {
      const int iw = (pc & 1) * 2 + (wid >> 1), ks = wid & 1;
      u32x4 xr[8];
      const float* xb = p.x + (size_t)(m * 256 + iw * 64 + (lane >> 3)) * KD + q * 1024 + pu * 64 + ks * 32 + (lane & 7) * 4;
#pragma unroll
      for (int i = 0; i < 8; ++i) {
        const float* ptr = xb + (size_t)(((((i >> 1) + q) & 3) * 16) + (i & 1) * 8) * KD;
        asm volatile("global_load_dwordx4 %0, %1, off" : "=v"(xr[i]) : "v"(ptr) : "memory");
      }
    if (wid == 3) {
      const int tile_ = m * 32 + ctile, dstr_ = ctile >> 2;
      const char* a_[16];
      a_[0] = myws + WS_XA + (size_t)(q * 2 + m) * 524288;  a_[1] = a_[0] + 262144;  a_[2] = a_[0] + 524288 - 128;
#pragma unroll
      for (int i = 0; i < 4; ++i) a_[3 + i] = myws + WS_XB + (size_t)(tile_ * 4 + i) * 3 * 16384 + 16384;
      { const int idx_ = ((ctile & 3) * 2 + m) * 4 + q;
        a_[7] = myws + WS_STG + (size_t)(dstr_ * 32 + idx_) * 4 * CHUNK;          a_[8] = a_[7] + (size_t)8 * 32 * 4 * CHUNK;
        a_[9] = p.ws[b & 7] + WS_STG + (size_t)(p.rank * 32 + (b >> 3)) * 4 * CHUNK; a_[10] = a_[9] + (size_t)8 * 32 * 4 * CHUNK; }
      a_[11] = (const char*)p.out[p.rank] + (size_t)((b & 7) * MS + ((b >> 5) & 1) * 256 + ((b >> 3) & 3) * 16) * NS;
      const int pr_ = (b >> 3) & 7;
      a_[12] = p.ws[pr_] + WS_GMAX;
      a_[13] = p.ws[dstr_] + WS_TFL; a_[14] = myws + WS_XFL + (size_t)tile_ * 512; a_[15] = myws + WS_GMAX;
#pragma unroll
      for (int i = 0; i < 16; ++i) asm volatile("s_load_dword %0, %1, 0x0" : "=s"(tw[i]) : "s"(a_[i]) : "memory");
    }
      WAIT_V(0);
      if (lane == 0) asm volatile("ds_add_u32 %0, %1" ::"v"(miscA + 16 * 4), "v"(1u) : "memory");
      STAMP(12);
#pragma unroll
      for (int i = 0; i < 8; ++i) asm volatile("" : "+v"(xr[i]));
      char* xa = myws + WS_XA + ((size_t)(((q * 2 + m) * 4 + iw) * 16 + pu) * 8 + ks * 4) * 1024;
      const bool odd = lane & 1;
      const int dl = (lane >> 3) + (odd ? 8 : 0) + 16 * ((lane & 7) >> 1);
#pragma unroll
      for (int i = 0; i < 8; i += 2) {
        const f32x4 a = __builtin_bit_cast(f32x4, xr[i]), c = __builtin_bit_cast(f32x4, xr[i + 1]);
        const unsigned a0 = cvt_pk(a[0], a[1]), a1 = cvt_pk(a[2], a[3]), c0 = cvt_pk(c[0], c[1]), c1 = cvt_pk(c[2], c[3]);
        const unsigned s0 = odd ? a0 : c0, s1 = odd ? a1 : c1;
        const unsigned r0 = __builtin_amdgcn_mov_dpp(s0, 0xB1, 0xF, 0xF, true), r1 = __builtin_amdgcn_mov_dpp(s1, 0xB1, 0xF, 0xF, true);
        const u32x4 o = odd ? (u32x4){r0, r1, c0, c1} : (u32x4){a0, a1, r0, r1};
        st_loc128(xa + (i >> 1) * 1024 + dl * 16, o);
      }
      if (lane == 0) asm volatile("ds_add_u32 %0, %1" ::"v"(miscA + 25 * 4), "v"(1u) : "memory");
      STAMP(13);
      WAIT_V(0);
      call = __builtin_amdgcn_readfirstlane(call);
      lds_put(miscA + (20 + wid) * 4, 1);
      if (wid == 0) {
        while (lds_min4(miscA + 20 * 4) == 0) __builtin_amdgcn_s_sleep(1);
        STAMP(14);
        if (lane == 0) { st_sys32(myws + WS_AFL + (q * 2 + m) * 128 + pc * 4, call); st_sys32(myws + WS_CNT + b * 128, call); misc[0] = call; }
        lds_put(miscA + 17 * 4, 4);
        const unsigned need = (3u << (2 * unit_of(0, m, n))) | (3u << (2 * unit_of(1, m, n)));
        SPIN_UNTIL(((have = (unsigned)__ballot(ld_sys32(myws + WS_AFL + (q * 2 + m) * 128 + (lane & 31) * 4) >= call)) & need) == need);
        if (dead) have = ~0u;
        lds_put(miscA + 19 * 4, have);
        lds_put(miscA + 20 * 4 - 8, 1);
      } else {
        if (wid == 3) asm volatile("s_waitcnt lgkmcnt(0)" : "+s"(tw[0]), "+s"(tw[1]), "+s"(tw[2]), "+s"(tw[3]), "+s"(tw[4]), "+s"(tw[5]), "+s"(tw[6]), "+s"(tw[7]),
                                   "+s"(tw[8]), "+s"(tw[9]), "+s"(tw[10]), "+s"(tw[11]), "+s"(tw[12]), "+s"(tw[13]), "+s"(tw[14]), "+s"(tw[15]) :: "memory");
        while (lds_get(miscA + 18 * 4) == 0) __builtin_amdgcn_s_sleep(1);
        have = lds_get(miscA + 19 * 4);
        call = lds_get(miscA);
      }
      STAMP(1);
    }
#pragma unroll
    for (int i = 0; i < 4; ++i)
#pragma unroll
      for (int nb = 0; nb < 8; ++nb) acc[i][nb] = (f32x4){0.f, 0.f, 0.f, 0.f};
    const char* xa = myws + WS_XA + (size_t)((q * 2 + m) * 4 + wid) * 16 * 8192;
    const unsigned xv0 = lane * 16, xv1 = xv0 + 4096;
    unsigned long long utab = 0;
#pragma unroll
    for (int t = 0; t < NU; ++t) utab |= (unsigned long long)unit_of(t, m, n) << (4 * t);
#define XADDR(T) (xa + (size_t)((unsigned)(utab >> (4 * (T))) & 15u) * 8192)
    u32x4 X0[2][4], X1[2][4];
#define LX(X, KS, I, A) TM3(c_lx, LX_(X, KS, I, A))
#define LX_(X, KS, I, A) asm volatile("global_load_dwordx4 %0, %1, %2 offset:%3 sc0 sc1" : "+v"(X[KS][I]) : "v"((KS) ? xv1 : xv0), "s"(A), "n"((I) * 1024) : "memory")
#define TM3(ACCU, STMT) do { STMT; } while (0)
#define WAITX_(X, KS, N) asm volatile("s_waitcnt vmcnt(%4)" : "+v"(X[KS][0]), "+v"(X[KS][1]), "+v"(X[KS][2]), "+v"(X[KS][3]) : "n"(N) : "memory")
#define WAITX(X, KS, N) TM3(c_wx, WAITX_(X, KS, N))
#pragma unroll
    for (int ks = 0; ks < 2; ++ks)
#pragma unroll
      for (int i = 0; i < 4; ++i) {
        X0[ks][i] = (u32x4){0, 0, 0, 0}; X1[ks][i] = (u32x4){0, 0, 0, 0};
      }
    const char* xn = XADDR(0);
    LX(X0, 0, 0, xn); LX(X0, 0, 1, xn); LX(X0, 0, 2, xn); LX(X0, 0, 3, xn);
    LX(X0, 1, 0, xn); LX(X0, 1, 1, xn); LX(X0, 1, 2, xn); LX(X0, 1, 3, xn);
    xn = XADDR(1);
    LX(X1, 0, 0, xn); LX(X1, 0, 1, xn); LX(X1, 0, 2, xn); LX(X1, 0, 3, xn);
    if (wid == 3) lds_put(miscA + 24 * 4, 1);
    unsigned BA0[2][2], BA1[2][2];
    {
      const int gg = lane >> 4, q4 = (lane & 15) >> 2, pp = lane & 3;
#pragma unroll
      for (int t = 0; t < 2; ++t)
#pragma unroll
        for (int cp = 0; cp < 2; ++cp) {
          BA0[t][cp] = (miscA - LDS_MISC) + 2048 * gg + 64 * (4 * t + q4) + 16 * ((2 * cp + (pp >> 1)) ^ ((2 * gg + t) & 3)) + 8 * (pp & 1);
          BA1[t][cp] = BA0[t][cp];
        }
    }
    bf16x8 W0[2], W1[2], W2[2], W3[2];
#define RDW(BA, C, T, KS, CP) __builtin_amdgcn_ds_read_tr16_b64_v4i16((LAS bf16x4*)(size_t)(BA[T][C] + (KS) * 8192 + (CP) * 512))
#define JOINW(lo, hi) ((bf16x8){lo[0], lo[1], lo[2], lo[3], hi[0], hi[1], hi[2], hi[3]})
#define MF(ACC, W, X) asm volatile("v_mfma_f32_16x16x32_bf16 %0, %1, %2, %0" : "+v"(ACC) : "v"(W), "v"(X))
#define SB0() __builtin_amdgcn_sched_barrier(0)
#define LOADW(W, BA, KS, CP) do { \
      { const bf16x4 lo = RDW(BA, 0, 0, KS, CP), hi = RDW(BA, 0, 1, KS, CP); W[0] = JOINW(lo, hi); } \
      { const bf16x4 lo = RDW(BA, 1, 0, KS, CP), hi = RDW(BA, 1, 1, KS, CP); W[1] = JOINW(lo, hi); } } while (0)
#define STEP(X, KS, W, CP, WN, BA, NKS, NCP, H1, H2, H3) do { \
      MF(acc[0][(CP) * 2], W[0], X[KS][0]); SB0(); \
      const bf16x4 l0 = RDW(BA, 0, 0, NKS, NCP); SB0(); \
      MF(acc[1][(CP) * 2], W[0], X[KS][1]); SB0(); H1; SB0(); MF(acc[2][(CP) * 2], W[0], X[KS][2]); SB0(); \
      const bf16x4 h0 = RDW(BA, 0, 1, NKS, NCP); SB0(); \
      MF(acc[3][(CP) * 2], W[0], X[KS][3]); MF(acc[0][(CP) * 2 + 1], W[1], X[KS][0]); SB0(); \
      const bf16x4 l1 = RDW(BA, 1, 0, NKS, NCP); SB0(); \
      MF(acc[1][(CP) * 2 + 1], W[1], X[KS][1]); SB0(); H2; SB0(); MF(acc[2][(CP) * 2 + 1], W[1], X[KS][2]); SB0(); \
      const bf16x4 h1 = RDW(BA, 1, 1, NKS, NCP); SB0(); \
      MF(acc[3][(CP) * 2 + 1], W[1], X[KS][3]); SB0(); H3; SB0(); \
      WN[0] = JOINW(l0, h0); WN[1] = JOINW(l1, h1); } while (0)
#define STEPL(X, KS, W, CP) do { _Pragma("unroll") for (int c = 0; c < 2; ++c) _Pragma("unroll") for (int i = 0; i < 4; ++i) MF(acc[i][(CP) * 2 + c], W[c], X[KS][i]); } while (0)
#define NEEDX(U) do { if (__builtin_expect(!((have >> (2 * (U) + (wid >> 1))) & 1u), 0)) { \
        SPIN_UNTIL(((have = (unsigned)__ballot(ld_sys32(myws + WS_AFL + (q * 2 + m) * 128 + (lane & 31) * 4) >= call)) >> (2 * (U) + (wid >> 1))) & 1u); \
        if (dead) have = ~0u; } } while (0)
#define NOP() do {} while (0)
#define VADD(D, S, K) asm volatile("v_add_u32 %0, %2, %1" : "=v"(D) : "v"(S), "s"(K))
#define UNIT(X, XO, BA, BN, T, L1, L0, N0, N4, LAST) do { \
      int d_; unsigned u_; TR(trc, (T)); \
      STEP(X, 0, W0, 0, W2, BA, 0, 2, if (L1) LX(XO, 1, 0, xn), if (L1) LX(XO, 1, 1, xn), d_ = (((T) + 1) & (NB - 1)) ? B_SLOT : -(NB - 1) * B_SLOT); \
      STEP(X, 0, W1, 1, W3, BA, 0, 3, if (L1) LX(XO, 1, 2, xn), if (L1) LX(XO, 1, 3, xn), u_ = (unsigned)(utab >> (4 * (((T) + 2) & 15))) & 15u); \
      STEP(X, 0, W2, 2, W0, BA, 1, 0, VADD(BN[0][0], BA[0][0], d_), VADD(BN[0][1], BA[0][1], d_), NOP()); \
      STEP(X, 0, W3, 3, W1, BA, 1, 1, VADD(BN[1][0], BA[1][0], d_), VADD(BN[1][1], BA[1][1], d_), WAITX(X, 1, N4)); \
      STEP(X, 1, W0, 0, W2, BA, 1, 2, if (L0) xn = xa + (size_t)u_ * 8192, if (L0) NEEDX(u_), NOP()); \
      STEP(X, 1, W1, 1, W3, BA, 1, 3, NOP(), NOP(), NOP()); \
      if (!(LAST)) { \
        TR(trc, 16 + (T)); asm volatile("s_barrier" ::: "memory"); TR(trc, 32 + (T)); SB0();        \
        STEP(X, 1, W2, 2, W0, BN, 0, 0, if (L0) LX(X, 0, 0, xn), if (L0) LX(X, 0, 1, xn), NOP()); \
        STEP(X, 1, W3, 3, W1, BN, 0, 1, if (L0) LX(X, 0, 2, xn), if (L0) LX(X, 0, 3, xn), WAITX(XO, 0, N0)); } \
      else { STEPL(X, 1, W2, 2); STEPL(X, 1, W3, 3); } } while (0)
    if (CPRIO) __builtin_amdgcn_s_setprio(CPRIO);
    unsigned trc = 0;
    bar();
    LOADW(W0, BA0, 0, 0); LOADW(W1, BA0, 0, 1);
    WAITX(X0, 0, 8);
    SB0();
    STAMP(2);
    for (int t = 0; t < NU - 2; t += 2) { UNIT(X0, X1, BA0, BA1, t, true, true, 8, 8, false); UNIT(X1, X0, BA1, BA0, t + 1, true, true, 8, 8, false); }
    UNIT(X0, X1, BA0, BA1, NU - 2, true, false, 4, 8, false);
    UNIT(X1, X0, BA1, BA0, NU - 1, false, false, 0, 0, true);
#undef LX
#undef XADDR
#undef NOP
#undef NEEDX
#undef VADD
#undef WAITX
#undef WAITX_
#undef LX_
#undef TM3
#undef LOADW
#undef RDW
#undef JOINW
#undef MF
#undef SB0
#undef STEP
#undef STEPL
#undef UNIT
    asm volatile("s_nop 15\n\ts_nop 15" ::: "memory");
    STAMP(3);
  } else {
    const int pw = wid - 4;
    const char* wb = (const char*)(p.w + (size_t)(q * 1024) * ND + ctile * 128 + pw * 32);
    const unsigned wvo = (unsigned)((lane >> 3) * ND * 4 + (lane & 7) * 16);
    u32x4 rb0[8], rb1[8];
    const int woff0 = 512 * pw + 64 * (lane >> 3) + 16 * (((lane & 7) >> 1) ^ (lane >> 5)) + 8 * (lane & 1);
    const int woff1 = 512 * pw + 64 * (lane >> 3) + 16 * (((lane & 7) >> 1) ^ (2 | (lane >> 5))) + 8 * (lane & 1);
    while (lds_get(miscA + 25 * 4) < 4) __builtin_amdgcn_s_sleep(1);
    issueB(rb0, wb, wvo, unit_of(0, m, n));
    unsigned trp = 0;
    WAIT_V(0); writeB(rb0, smem, woff0, woff1, 0);
    bar();
    issueB(rb1, wb, wvo, unit_of(1, m, n));
    issueB(rb0, wb, wvo, unit_of(2, m, n));
    WAIT_V(8); writeB(rb1, smem, woff0, woff1, 1);
    issueB(rb1, wb, wvo, unit_of(3, m, n));
    bar();
#define PSTEP(s, RB, MORE, NW) do { \
      TR(trp, (s)); WAIT_V(NW); TR(trp, 16 + (s)); \
      writeB(RB, smem, woff0, woff1, (s) & (NB - 1)); \
      if (MORE) { TR(trp, 32 + (s)); issueB(RB, wb, wvo, unit_of((s) + 2, m, n)); TR(trp, 48 + (s)); } \
      bar(); } while (0)
    for (int s = 2; s < NU - 2; s += 2) { PSTEP(s, rb0, true, 8); PSTEP(s + 1, rb1, true, 8); }
    PSTEP(NU - 2, rb0, false, 8); PSTEP(NU - 1, rb1, false, 0);
#undef PSTEP
  }
  bar();
  asm volatile("" : "+v"(tid));
  lane = tid & 63;

  const int tile = m * 32 + ctile;
  call = misc[0];
  const unsigned par = call & 1;
  const unsigned ldsb = miscA - LDS_MISC;
  if (wid < 4) {
#pragma unroll
    for (int i = 1; i < 4; ++i) {
      char* dst = myws + WS_XB + ((size_t)(tile * 4 + ((q + i) & 3)) * 3 + (i - 1)) * 16384 + wid * 4096 + lane * 16;
#pragma unroll
      for (int e = 0; e < 4; ++e) {
        u32x4 o = {cvt_pk(acc[i][2 * e][0], acc[i][2 * e][1]), cvt_pk(acc[i][2 * e][2], acc[i][2 * e][3]),
                   cvt_pk(acc[i][2 * e + 1][0], acc[i][2 * e + 1][1]), cvt_pk(acc[i][2 * e + 1][2], acc[i][2 * e + 1][3])};
        st_loc128(dst + e * 1024, o);
      }
    }
    u32x4 pr[3][4];
    char* src = myws + WS_XB + (size_t)(tile * 4 + q) * 3 * 16384 + wid * 4096 + lane * 16;
    auto fetch = [&]() {
#pragma unroll
      for (int i = 0; i < 3; ++i)
#pragma unroll
        for (int e = 0; e < 4; ++e) asm volatile("global_load_dwordx4 %0, %1, off sc0 sc1" : "=v"(pr[i][e]) : "v"(src + i * 16384 + e * 1024) : "memory");
      asm volatile("s_waitcnt vmcnt(0)" : "+v"(pr[0][0]), "+v"(pr[0][1]), "+v"(pr[0][2]), "+v"(pr[0][3]), "+v"(pr[1][0]), "+v"(pr[1][1]), "+v"(pr[1][2]), "+v"(pr[1][3]),
                   "+v"(pr[2][0]), "+v"(pr[2][1]), "+v"(pr[2][2]), "+v"(pr[2][3]) :: "memory");
      unsigned mn = 0xffffffffu;
#pragma unroll
      for (int i = 0; i < 3; ++i)
#pragma unroll
        for (int e = 0; e < 4; ++e) mn = min(min(mn, min(pr[i][e][0], pr[i][e][1])), min(pr[i][e][2], pr[i][e][3]));
      return __all(mn != 0u) != 0; };
    WAIT_V(0);
    if (lane < 3) st_sys32(myws + WS_XFL + (size_t)(tile * 4 + ((q + lane + 1) & 3)) * 128 + (lane * 4 + wid) * 4, call);
    SPIN_UNTIL(__all(ld_sys32(myws + WS_XFL + (size_t)(tile * 4 + q) * 128 + ((lane < 3 ? lane : 0) * 4 + wid) * 4) >= call));
    fetch();
    STAMP(4);
#pragma unroll
    for (int i = 0; i < 3; ++i)
#pragma unroll
      for (int e = 0; e < 4; ++e)
#pragma unroll
        for (int d = 0; d < 4; ++d) {
          acc[0][2 * e + (d >> 1)][(d & 1) * 2] += __uint_as_float(pr[i][e][d] << 16);
          acc[0][2 * e + (d >> 1)][(d & 1) * 2 + 1] += __uint_as_float(pr[i][e][d] & 0xffff0000u);
        }
    float mx = 0.f;
#pragma unroll
    for (int nb = 0; nb < 8; ++nb)
#pragma unroll
      for (int r = 0; r < 4; ++r) { const float v = fmaxf(acc[0][nb][r], 0.f); acc[0][nb][r] = v; mx = fmaxf(mx, v); }
    mx = __uint_as_float(wave_max_u32(__float_as_uint(mx)));
    if (lane == 0) {
      unsigned old;
      asm volatile("ds_max_u32 %1, %2\n\tds_add_rtn_u32 %0, %3, %4\n\ts_waitcnt lgkmcnt(0)" : "=v"(old) : "v"(miscA + 4 * 4), "v"(__float_as_uint(mx)), "v"(miscA + 5 * 4), "v"(1u) : "memory");
      if (old == 3) st_sys64(myws + WS_LMAX + par * 2048 + b * 8, ((unsigned long long)call << 32) | misc[4]);
    }
    STAMP(5);
    const float inv16 = mx > 0.f ? 65535.0f / mx : 0.f;
    unsigned code[32], mask = 0;
#pragma unroll
    for (int i = 0; i < 32; ++i) { code[i] = min((unsigned)__builtin_fmaf(acc[0][i >> 2][i & 3], inv16, 0.5f), 65535u); mask |= (code[i] != 0u ? 1u : 0u) << i; }
    const unsigned cnt = __builtin_popcount(mask);
    const unsigned incl = wave_scan_u32(cnt);
    const unsigned total = __builtin_amdgcn_readlane(incl, 63);
    const unsigned sb_ = ldsb + wid * CHUNK;
    *(LAS unsigned*)(size_t)(sb_ + 16 + lane * 4) = mask;
    if (lane == 0) *(LAS u32x4*)(size_t)sb_ = (u32x4){total, __float_as_uint(mx), call, 0u};
    const unsigned off = sb_ + 272 + 2 * (incl - cnt);
    const unsigned dummy = ldsb + 4 * CHUNK + lane * 2;
#pragma unroll
    for (int i = 0; i < 32; ++i) *(LAS unsigned short*)(size_t)(code[i] ? off + 2 * __builtin_popcount(mask & ((1u << i) - 1u)) : dummy) = (unsigned short)code[i];
    const int dstr = ctile >> 2, idx = ((ctile & 3) * 2 + m) * 4 + q;
    char* out_ = myws + WS_STG + ((size_t)((par * 8 + dstr) * 32 + idx) * 4 + wid) * CHUNK;
    const unsigned units = (272 + 2 * total + 15) >> 4;
    asm volatile("s_waitcnt lgkmcnt(0)" ::: "memory");
    STAMP(9);
    for (unsigned u = lane; u < ((units + 63) & ~63u); u += 64)
      if (u < units) st_wt128(out_ + u * 16, *(const LAS u32x4*)(size_t)(sb_ + u * 16));
    STAMP(10);
    WAIT_V(0);
    if (lane == 0) { st_sys64(p.ws[dstr] + WS_TFL + ((size_t)((par * 8 + p.rank) * 32 + idx) * 4 + wid) * 8, ((unsigned long long)call << 32) | units);
      asm volatile("ds_add_u32 %0, %1" ::"v"(miscA + 27 * 4), "v"(1u) : "memory"); }
    STAMP(7);
  } else {
    const int pw = wid - 4, srcr = b & 7, idx = b >> 3;
    const char* in_ = p.ws[srcr] + WS_STG + ((size_t)((par * 8 + p.rank) * 32 + idx) * 4 + pw) * CHUNK;
    const unsigned rb_ = ldsb + 32768 + pw * CHUNK;
    float val[32]; float cscale = 0.f;
    bool gdone = !(b == p.rank && wid == 7), tdone = false, adone = wid != 4;
    while (lds_get(miscA + 27 * 4) < 4) __builtin_amdgcn_s_sleep(4);
    STAMP(11);
    auto progress = [&]() {
      if (!gdone) {
        bool ok = true; unsigned lm = 0;
#pragma unroll
        for (int i = 0; i < 4; ++i) { const unsigned long long gv = ld_sys64(myws + WS_LMAX + par * 2048 + (i * 64 + lane) * 8); ok &= (unsigned)(gv >> 32) == call; lm = max(lm, (unsigned)gv); }
        if (__all(ok)) {
          lm = wave_max_u32(lm);
          st_sys64(p.ws[lane >> 3] + WS_GMAX + par * 1024 + (lane & 7) * 128 + p.rank * 8, ((unsigned long long)call << 32) | lm);
          gdone = true;
        }
      }
      unsigned long long tf = 0;
      if (!tdone) tf = ld_sys64(myws + WS_TFL + ((size_t)((par * 8 + srcr) * 32 + idx) * 4 + pw) * 8);
      if (!tdone && __builtin_amdgcn_readfirstlane((unsigned)(tf >> 32)) == call) {
        STAMP(9);
        const unsigned units = min((unsigned)__builtin_amdgcn_readfirstlane((unsigned)tf), (unsigned)(CHUNK / 16));
        u32x4 r[5];
#pragma unroll
        for (int k = 0; k < 5; ++k) { r[k] = (u32x4){0u, 0u, 0u, 0u}; if (k * 64 + lane < units) asm volatile("global_load_dwordx4 %0, %1, off sc0 sc1" : "+v"(r[k]) : "v"(in_ + (k * 64 + lane) * 16) : "memory"); }
        asm volatile("s_waitcnt vmcnt(0)" : "+v"(r[0]), "+v"(r[1]), "+v"(r[2]), "+v"(r[3]), "+v"(r[4]) :: "memory");
        STAMP(10);
#pragma unroll
        for (int k = 0; k < 5; ++k) if (k * 64 + lane < CHUNK / 16) *(LAS u32x4*)(size_t)(rb_ + (k * 64 + lane) * 16) = r[k];
        asm volatile("s_waitcnt lgkmcnt(0)" ::: "memory");
        const unsigned mask = *(const LAS unsigned*)(size_t)(rb_ + 16 + lane * 4);
        cscale = __uint_as_float(*(const LAS unsigned*)(size_t)(rb_ + 4));
        const unsigned cnt = __builtin_popcount(mask);
        const unsigned off = rb_ + 272 + 2 * (wave_scan_u32(cnt) - cnt);
        unsigned short cv[32];
#pragma unroll
        for (int i = 0; i < 32; ++i) cv[i] = *(const LAS unsigned short*)(size_t)(off + 2 * __builtin_popcount(mask & ((1u << i) - 1u)));
#pragma unroll
        for (int i = 0; i < 32; ++i) val[i] = ((mask >> i) & 1u) ? (float)cv[i] : 0.f;
        tdone = true;
      }
      if (!adone) {
        const unsigned long long gv = ld_sys64(myws + WS_GMAX + par * 1024 + (b & 7) * 128 + (lane & 7) * 8);
        if (__all((unsigned)(gv >> 32) == call)) { lds_put(miscA + 1 * 4, wave_max_u32((unsigned)gv)); lds_put(miscA + 26 * 4, 1); adone = true; }
      }
      return gdone && tdone && adone; };
    SPIN_UNTIL(progress());
    STAMP(6);
    if (wid == 4) { if (!adone) lds_put(miscA + 26 * 4, 1); }
    else { while (lds_get(miscA + 26 * 4) == 0) __builtin_amdgcn_s_sleep(1); }
    const float amax = __uint_as_float(lds_get(miscA + 1 * 4));
    const bool bad = dead || __builtin_amdgcn_readfirstlane(ld_sys32(myws + WS_TMO)) == call;
    STAMP(7);
    const float cq = amax > 0.f ? cscale * (127.0f / 65535.0f) / amax : 0.f;
    const unsigned tb_ = ldsb + 65536 + pw * 2304;
#pragma unroll
    for (int nb = 0; nb < 8; ++nb) {
      unsigned v = 0;
#pragma unroll
      for (int e = 0; e < 4; ++e) { int qv = (int)__builtin_rintf(val[nb * 4 + e] * cq); qv = min(qv, 127); v |= (unsigned)(qv & 255) << (8 * e); }
      *(LAS unsigned*)(size_t)(tb_ + (lane & 15) * 144 + nb * 16 + (lane >> 4) * 4) = bad ? 0x80808080u : v;
    }
    asm volatile("s_waitcnt lgkmcnt(0)" ::: "memory");
    signed char* ob = p.out[p.rank] + (size_t)(srcr * MS + ((idx >> 2) & 1) * 256 + pw * 64 + (idx & 3) * 16) * NS + (idx >> 3) * 128;
#pragma unroll
    for (int k = 0; k < 2; ++k) { const int r = k * 8 + (lane >> 3);
      *(u32x4*)(ob + (size_t)r * NS + (lane & 7) * 16) = *(const LAS u32x4*)(size_t)(tb_ + r * 144 + (lane & 7) * 16); }
  }
  STAMP(8);
}

static unsigned long long* g_stamps = nullptr;

extern "C" void kernel_launch(int rank, int nranks, ncclComm_t comm, hipStream_t stream,
                              void* const* d_in, const int* in_sizes, int n_in,
                              void* d_out, int out_size, const int* in_dtypes, int out_dtype,
                              void* const* d_out_all, void* const* d_ws_all, size_t ws_bytes) {
  if (nranks != NR || ws_bytes < WS_END || n_in < 2) { fprintf(stderr, "bad config\n"); return; }
  if (in_sizes[0] != MS * KD || in_sizes[1] != KD * ND || out_size != 4096 * NS) { fprintf(stderr, "bad sizes\n"); return; }
  HIP_CHECK(hipFuncSetAttribute((const void*)gemm_a2a, hipFuncAttributeMaxDynamicSharedMemorySize, LDS_BYTES));
  Params p;
  p.x = (const float*)d_in[0]; p.w = (const float*)d_in[1];
  for (int i = 0; i < NR; ++i) { p.ws[i] = (char*)d_ws_all[i]; p.out[i] = (signed char*)d_out_all[i]; }
  p.stamps = g_stamps;
  p.rank = rank; p.pad = 0;
  hipLaunchKernelGGL(gemm_a2a, dim3(NWG), dim3(NTHR), LDS_BYTES, stream, p);
  HIP_CHECK(hipGetLastError());
}
```

```cpp
#include <hip/hip_runtime.h>
#include <rccl/rccl.h>
#include <cstdio>
#include <cstdint>

#define GMODE 5
#define XMODE 0
#define NTOK 2
#define OPT_ADD 1
#define OPT_PACK 1
#define OPT_EXP 1
#define RMAP 0
#define RSRC(b) ((b) & 7)
#define RIDX(b) ((b) >> 3)
#define RGATH(r) (r)
#define PSLEEP 1
#define CPRIO 0
#define HIP_CHECK(expr) do { hipError_t _e = (expr); if (_e != hipSuccess) { \
    fprintf(stderr, "HIP error %s at %s:%d\n", hipGetErrorString(_e), __FILE__, __LINE__); \
    return; } } while (0)

typedef unsigned u32x4 __attribute__((ext_vector_type(4)));
typedef unsigned u32x2 __attribute__((ext_vector_type(2)));
typedef float f32x4 __attribute__((ext_vector_type(4)));
typedef float f32x2 __attribute__((ext_vector_type(2)));
typedef short bf16x8 __attribute__((ext_vector_type(8)));
typedef short bf16x4 __attribute__((ext_vector_type(4)));
typedef __bf16 bf16x2_t __attribute__((ext_vector_type(2)));
#define LAS __attribute__((address_space(3)))

constexpr int NR = 8;
constexpr int MS = 512;
constexpr int KD = 4096, ND = 4096;
constexpr int NS = 512;
constexpr int NWG = 256, NTHR = 512;
constexpr int NU = 16;
constexpr unsigned long long DEADLINE = 40000000ull;

constexpr int B_SLOT = 16384, NB = 8;
constexpr int LDS_MISC = NB * B_SLOT, LDS_BYTES = LDS_MISC + 256;

constexpr size_t WS_CNT  = 0;
constexpr size_t WS_AFL  = WS_CNT + 256 * 128;
constexpr size_t WS_XFL  = WS_AFL + 8 * 128;
constexpr size_t WS_LMAX = WS_XFL + 256 * 128;
constexpr size_t WS_GMAX = WS_LMAX + 2 * 256 * 8;
constexpr size_t WS_DONE = WS_GMAX + 2 * 8 * 128;
constexpr size_t WS_TFL  = WS_DONE + 8 * 128;
constexpr size_t WS_TMO  = WS_TFL + 16384;
constexpr size_t WS_ALN  = WS_TMO + 128;
constexpr size_t WS_XA   = 1 << 20;
constexpr size_t WS_XB   = WS_XA + (4 << 20);
constexpr int CHUNK = 4608;
constexpr size_t WS_STG  = WS_XB + (size_t)64 * 4 * 3 * 16384;
constexpr size_t WS_END  = WS_STG + (size_t)2 * 8 * 32 * 4 * CHUNK;
static_assert(WS_ALN + 8 * 128 <= WS_XA, "control area");

struct Params {
  const float* x; const float* w;
  char* ws[NR]; signed char* out[NR];
  unsigned long long* stamps;
  int rank; int pad;
};

__device__ __forceinline__ unsigned cvt_pk(float lo, float hi) {
  f32x2 v = {lo, hi};
  return __builtin_bit_cast(unsigned, __builtin_convertvector(v, bf16x2_t));
}
__device__ __forceinline__ unsigned long long ld_sys64(const void* p) { return __hip_atomic_load((const unsigned long long*)p, __ATOMIC_RELAXED, __HIP_MEMORY_SCOPE_SYSTEM); }
__device__ __forceinline__ unsigned ld_sys32(const void* p) { return __hip_atomic_load((const unsigned*)p, __ATOMIC_RELAXED, __HIP_MEMORY_SCOPE_SYSTEM); }
__device__ __forceinline__ void st_sys64(void* p, unsigned long long v) { __hip_atomic_store((unsigned long long*)p, v, __ATOMIC_RELAXED, __HIP_MEMORY_SCOPE_SYSTEM); }
__device__ __forceinline__ void st_sys32(void* p, unsigned v) { __hip_atomic_store((unsigned*)p, v, __ATOMIC_RELAXED, __HIP_MEMORY_SCOPE_SYSTEM); }
#define LSC "sc0 sc1"
__device__ __forceinline__ void st_loc128(void* p, u32x4 v) { asm volatile("global_store_dwordx4 %0, %1, off " LSC "\n\ts_nop 1" ::"v"(p), "v"(v) : "memory"); }
__device__ __forceinline__ void st_wt128(void* p, u32x4 v) { asm volatile("global_store_dwordx4 %0, %1, off sc0 sc1\n\ts_nop 1" ::"v"(p), "v"(v) : "memory"); }
#define WAIT_V(n) asm volatile("s_waitcnt vmcnt(%0)" ::"n"(n) : "memory")
__device__ __forceinline__ void bar() { asm volatile("s_waitcnt lgkmcnt(0)\n\ts_barrier" ::: "memory"); }
__device__ __forceinline__ void lds_put(unsigned addr, unsigned v) { asm volatile("ds_write_b32 %0, %1" ::"v"(addr), "v"(v) : "memory"); }
__device__ __forceinline__ unsigned lds_get(unsigned addr) { unsigned r; asm volatile("ds_read_b32 %0, %1\n\ts_waitcnt lgkmcnt(0)" : "=v"(r) : "v"(addr) : "memory"); return __builtin_amdgcn_readfirstlane(r); }
__device__ __forceinline__ unsigned lds_min4(unsigned addr) {
  u32x4 r; asm volatile("ds_read_b128 %0, %1\n\ts_waitcnt lgkmcnt(0)" : "=v"(r) : "v"(addr) : "memory");
  return __builtin_amdgcn_readfirstlane(min(min(r.x, r.y), min(r.z, r.w)));
}
#define SPIN_UNTIL(COND) do { const unsigned long long t0_ = __builtin_amdgcn_s_memrealtime(); unsigned it_ = 0; \
    while (!(COND)) { \
      if ((++it_ & 63u) == 0u && (ld_sys32(myws + WS_TMO) == call || __builtin_amdgcn_s_memrealtime() - t0_ > DEADLINE)) { dead = true; break; } \
      __builtin_amdgcn_s_sleep(1); } \
    if (dead) { if (lane == 0) st_sys32(myws + WS_TMO, call); if (lane < 16) st_sys32(myws + WS_GMAX + lane * 128 + 64, call); } } while (0)
#define STAMP(i) do {} while (0)
#define RSTAMP(i) do {} while (0)
#define TR(V, IDX) do {} while (0)

__device__ __forceinline__ int unit_of(int t, int m, int n) {
  int v = t;
  if (m) v = (t == 0) ? 1 : (t == 15) ? 14 : ((t & 1) ? t + 2 : t - 2);
  return (v + n) & 15;
}

__device__ __forceinline__ void issueB(u32x4 (&rb)[8], const char* wbase, unsigned voff, int unit) {
  const char* src = wbase + (size_t)unit * 64 * ND * 4;
#pragma unroll
  for (int i = 0; i < 8; ++i) asm volatile("global_load_dwordx4 %0, %1, %2" : "=v"(rb[i]) : "v"(voff), "s"(src + (size_t)i * 8 * ND * 4) : "memory");
}
__device__ __forceinline__ void writeB(u32x4 (&rb)[8], char* smem, int woff0, int woff1, int slot) {
  LAS char* B = (LAS char*)smem + slot * B_SLOT;
#pragma unroll
  for (int i = 0; i < 8; ++i) {
    asm volatile("" : "+v"(rb[i]));
    const f32x4 v = __builtin_bit_cast(f32x4, rb[i]);
    *(LAS u32x2*)(B + ((i & 1) ? woff1 : woff0) + i * 2048) = (u32x2){cvt_pk(v[0], v[1]), cvt_pk(v[2], v[3])};
  }
}

__device__ __forceinline__ unsigned wave_max_u32(unsigned v) {
  v = max(v, (unsigned)__builtin_amdgcn_update_dpp(0u, v, 0x111, 0xf, 0xf, false));
  v = max(v, (unsigned)__builtin_amdgcn_update_dpp(0u, v, 0x112, 0xf, 0xf, false));
  v = max(v, (unsigned)__builtin_amdgcn_update_dpp(0u, v, 0x114, 0xf, 0xf, false));
  v = max(v, (unsigned)__builtin_amdgcn_update_dpp(0u, v, 0x118, 0xf, 0xf, false));
  v = max(v, (unsigned)__builtin_amdgcn_update_dpp(0u, v, 0x142, 0xa, 0xf, false));
  v = max(v, (unsigned)__builtin_amdgcn_update_dpp(0u, v, 0x143, 0xc, 0xf, false));
  return __builtin_amdgcn_readlane(v, 63);
}
__device__ __forceinline__ unsigned wave_scan_u32(unsigned v) {
  v += __builtin_amdgcn_update_dpp(0u, v, 0x111, 0xf, 0xf, false);
  v += __builtin_amdgcn_update_dpp(0u, v, 0x112, 0xf, 0xf, false);
  v += __builtin_amdgcn_update_dpp(0u, v, 0x114, 0xf, 0xf, false);
  v += __builtin_amdgcn_update_dpp(0u, v, 0x118, 0xf, 0xf, false);
  v += __builtin_amdgcn_update_dpp(0u, v, 0x142, 0xa, 0xf, false);
  v += __builtin_amdgcn_update_dpp(0u, v, 0x143, 0xc, 0xf, false);
  return v;
}

__global__ __launch_bounds__(NTHR, 2) void gemm_a2a(Params p) {
  extern __shared__ __attribute__((aligned(16))) char smem[];
  int tid = threadIdx.x, lane = tid & 63;
  const int wid = __builtin_amdgcn_readfirstlane(tid >> 6);
  const int b = blockIdx.x;
  const int g = b & 7, q = g & 3, h = g >> 2, j = b >> 3, m = j & 1, n = j >> 1;
  const int ctile = h * 16 + n;
  char* myws = p.ws[p.rank];
  volatile LAS unsigned* misc = (volatile LAS unsigned*)(smem + LDS_MISC);
  const unsigned miscA = (unsigned)(size_t)(LAS char*)smem + LDS_MISC;
  if (tid < 64) misc[tid] = 0;
  bar();
  STAMP(0);

  f32x4 acc[4][8];
  unsigned call = 0;
  bool dead = false;

  if (wid < 4) {
    if (tid == 0) { call = ld_sys32(myws + WS_CNT + b * 128) + 1; }
    unsigned have = 0;
    unsigned tw[16];
    const int pu = unit_of(0, m, n), pc = pu * 2 + h;
    {
      const int iw = (pc & 1) * 2 + (wid >> 1), ks = wid & 1;
      u32x4 xr[8];
      const float* xb = p.x + (size_t)(m * 256 + iw * 64 + (lane >> 3)) * KD + q * 1024 + pu * 64 + ks * 32 + (lane & 7) * 4;
#pragma unroll
      for (int i = 0; i < 8; ++i) {
        const float* ptr = xb + (size_t)(((((i >> 1) + q) & 3) * 16) + (i & 1) * 8) * KD;
        asm volatile("global_load_dwordx4 %0, %1, off" : "=v"(xr[i]) : "v"(ptr) : "memory");
      }
    if (wid == 3) {
      const int tile_ = m * 32 + ctile, dstr_ = ctile >> 2;
      const char* a_[16];
      a_[0] = myws + WS_XA + (size_t)(q * 2 + m) * 524288;  a_[1] = a_[0] + 262144;  a_[2] = a_[0] + 524288 - 128;
#pragma unroll
      for (int i = 0; i < 4; ++i) a_[3 + i] = myws + WS_XB + (size_t)(tile_ * 4 + i) * 3 * 16384 + 16384;
      { const int idx_ = ((ctile & 3) * 2 + m) * 4 + q;
        a_[7] = myws + WS_STG + (size_t)(dstr_ * 32 + idx_) * 4 * CHUNK;          a_[8] = a_[7] + (size_t)8 * 32 * 4 * CHUNK;
        a_[9] = p.ws[RSRC(b)] + WS_STG + (size_t)(p.rank * 32 + RIDX(b)) * 4 * CHUNK; a_[10] = a_[9] + (size_t)8 * 32 * 4 * CHUNK; }
      a_[11] = (const char*)p.out[p.rank] + (size_t)(RSRC(b) * MS + ((RIDX(b) >> 2) & 1) * 256 + (RIDX(b) & 3) * 16) * NS;
      const int pr_ = (b >> 3) & 7;
      a_[12] = p.ws[pr_] + WS_GMAX;
      a_[13] = p.ws[dstr_] + WS_TFL; a_[14] = myws + WS_XFL + (size_t)tile_ * 512; a_[15] = myws + WS_GMAX;
#pragma unroll
      for (int i = 0; i < 16; ++i) asm volatile("s_load_dword %0, %1, 0x0" : "=s"(tw[i]) : "s"(a_[i]) : "memory");
    }
      WAIT_V(0);
      if (lane == 0) asm volatile("ds_add_u32 %0, %1" ::"v"(miscA + 16 * 4), "v"(1u) : "memory");
      STAMP(12);
#pragma unroll
      for (int i = 0; i < 8; ++i) asm volatile("" : "+v"(xr[i]));
      char* xa = myws + WS_XA + ((size_t)(((q * 2 + m) * 4 + iw) * 16 + pu) * 8 + ks * 4) * 1024;
      const bool odd = lane & 1;
      const int dl = (lane >> 3) + (odd ? 8 : 0) + 16 * ((lane & 7) >> 1);
#pragma unroll
      for (int i = 0; i < 8; i += 2) {
        const f32x4 a = __builtin_bit_cast(f32x4, xr[i]), c = __builtin_bit_cast(f32x4, xr[i + 1]);
        const unsigned a0 = cvt_pk(a[0], a[1]), a1 = cvt_pk(a[2], a[3]), c0 = cvt_pk(c[0], c[1]), c1 = cvt_pk(c[2], c[3]);
        const unsigned s0 = odd ? a0 : c0, s1 = odd ? a1 : c1;
        const unsigned r0 = __builtin_amdgcn_mov_dpp(s0, 0xB1, 0xF, 0xF, true), r1 = __builtin_amdgcn_mov_dpp(s1, 0xB1, 0xF, 0xF, true);
        const u32x4 o = odd ? (u32x4){r0, r1, c0, c1} : (u32x4){a0, a1, r0, r1};
        st_loc128(xa + (i >> 1) * 1024 + dl * 16, o);
      }
      if (lane == 0) asm volatile("ds_add_u32 %0, %1" ::"v"(miscA + 25 * 4), "v"(1u) : "memory");
      STAMP(13);
      WAIT_V(0);
      call = __builtin_amdgcn_readfirstlane(call);
      lds_put(miscA + (20 + wid) * 4, 1);
      if (wid == 0) {
        while (lds_min4(miscA + 20 * 4) == 0) __builtin_amdgcn_s_sleep(1);
        STAMP(14);
        if (lane == 0) { st_sys32(myws + WS_AFL + (q * 2 + m) * 128 + pc * 4, call); st_sys32(myws + WS_CNT + b * 128, call); misc[0] = call; }
        lds_put(miscA + 17 * 4, 4);
        const unsigned need = (3u << (2 * unit_of(0, m, n))) | (3u << (2 * unit_of(1, m, n)));
        SPIN_UNTIL(((have = (unsigned)__ballot(ld_sys32(myws + WS_AFL + (q * 2 + m) * 128 + (lane & 31) * 4) >= call)) & need) == need);
        if (dead) have = ~0u;
        lds_put(miscA + 19 * 4, have);
        lds_put(miscA + 20 * 4 - 8, 1);
      } else {
        if (wid == 3) asm volatile("s_waitcnt lgkmcnt(0)" : "+s"(tw[0]), "+s"(tw[1]), "+s"(tw[2]), "+s"(tw[3]), "+s"(tw[4]), "+s"(tw[5]), "+s"(tw[6]), "+s"(tw[7]),
                                   "+s"(tw[8]), "+s"(tw[9]), "+s"(tw[10]), "+s"(tw[11]), "+s"(tw[12]), "+s"(tw[13]), "+s"(tw[14]), "+s"(tw[15]) :: "memory");
        while (lds_get(miscA + 18 * 4) == 0) __builtin_amdgcn_s_sleep(1);
        have = lds_get(miscA + 19 * 4);
        call = lds_get(miscA);
      }
      STAMP(1);
    }
#pragma unroll
    for (int i = 0; i < 4; ++i)
#pragma unroll
      for (int nb = 0; nb < 8; ++nb) acc[i][nb] = (f32x4){0.f, 0.f, 0.f, 0.f};
    const char* xa = myws + WS_XA + (size_t)((q * 2 + m) * 4 + wid) * 16 * 8192;
    const unsigned xv0 = lane * 16, xv1 = xv0 + 4096;
    unsigned long long utab = 0;
#pragma unroll
    for (int t = 0; t < NU; ++t) utab |= (unsigned long long)unit_of(t, m, n) << (4 * t);
#define XADDR(T) (xa + (size_t)((unsigned)(utab >> (4 * (T))) & 15u) * 8192)
    u32x4 X0[2][4], X1[2][4];
#define LX(X, KS, I, A) TM3(c_lx, LX_(X, KS, I, A))
#define LX_(X, KS, I, A) asm volatile("global_load_dwordx4 %0, %1, %2 offset:%3 sc0 sc1" : "+v"(X[KS][I]) : "v"((KS) ? xv1 : xv0), "s"(A), "n"((I) * 1024) : "memory")
#define TM3(ACCU, STMT) do { STMT; } while (0)
#define WAITX_(X, KS, N) asm volatile("s_waitcnt vmcnt(%4)" : "+v"(X[KS][0]), "+v"(X[KS][1]), "+v"(X[KS][2]), "+v"(X[KS][3]) : "n"(N) : "memory")
#define WAITX(X, KS, N) TM3(c_wx, WAITX_(X, KS, N))
#pragma unroll
    for (int ks = 0; ks < 2; ++ks)
#pragma unroll
      for (int i = 0; i < 4; ++i) {
        X0[ks][i] = (u32x4){0, 0, 0, 0}; X1[ks][i] = (u32x4){0, 0, 0, 0};
      }
    const char* xn = XADDR(0);
    LX(X0, 0, 0, xn); LX(X0, 0, 1, xn); LX(X0, 0, 2, xn); LX(X0, 0, 3, xn);
    LX(X0, 1, 0, xn); LX(X0, 1, 1, xn); LX(X0, 1, 2, xn); LX(X0, 1, 3, xn);
    xn = XADDR(1);
    LX(X1, 0, 0, xn); LX(X1, 0, 1, xn); LX(X1, 0, 2, xn); LX(X1, 0, 3, xn);
    if (wid == 3) lds_put(miscA + 24 * 4, 1);
    unsigned BA0[2][2], BA1[2][2];
    {
      const int gg = lane >> 4, q4 = (lane & 15) >> 2, pp = lane & 3;
#pragma unroll
      for (int t = 0; t < 2; ++t)
#pragma unroll
        for (int cp = 0; cp < 2; ++cp) {
          BA0[t][cp] = (miscA - LDS_MISC) + 2048 * gg + 64 * (4 * t + q4) + 16 * ((2 * cp + (pp >> 1)) ^ ((2 * gg + t) & 3)) + 8 * (pp & 1);
          BA1[t][cp] = BA0[t][cp];
        }
    }
    bf16x8 W0[2], W1[2], W2[2], W3[2];
#define RDW(BA, C, T, KS, CP) __builtin_amdgcn_ds_read_tr16_b64_v4i16((LAS bf16x4*)(size_t)(BA[T][C] + (KS) * 8192 + (CP) * 512))
#define JOINW(lo, hi) ((bf16x8){lo[0], lo[1], lo[2], lo[3], hi[0], hi[1], hi[2], hi[3]})
#define MF(ACC, W, X) asm volatile("v_mfma_f32_16x16x32_bf16 %0, %1, %2, %0" : "+v"(ACC) : "v"(W), "v"(X))
#define SB0() __builtin_amdgcn_sched_barrier(0)
#define LOADW(W, BA, KS, CP) do { \
      { const bf16x4 lo = RDW(BA, 0, 0, KS, CP), hi = RDW(BA, 0, 1, KS, CP); W[0] = JOINW(lo, hi); } \
      { const bf16x4 lo = RDW(BA, 1, 0, KS, CP), hi = RDW(BA, 1, 1, KS, CP); W[1] = JOINW(lo, hi); } } while (0)
#define STEP(X, KS, W, CP, WN, BA, NKS, NCP, H1, H2, H3) do { \
      MF(acc[0][(CP) * 2], W[0], X[KS][0]); SB0(); \
      const bf16x4 l0 = RDW(BA, 0, 0, NKS, NCP); SB0(); \
      MF(acc[1][(CP) * 2], W[0], X[KS][1]); SB0(); H1; SB0(); MF(acc[2][(CP) * 2], W[0], X[KS][2]); SB0(); \
      const bf16x4 h0 = RDW(BA, 0, 1, NKS, NCP); SB0(); \
      MF(acc[3][(CP) * 2], W[0], X[KS][3]); MF(acc[0][(CP) * 2 + 1], W[1], X[KS][0]); SB0(); \
      const bf16x4 l1 = RDW(BA, 1, 0, NKS, NCP); SB0(); \
      MF(acc[1][(CP) * 2 + 1], W[1], X[KS][1]); SB0(); H2; SB0(); MF(acc[2][(CP) * 2 + 1], W[1], X[KS][2]); SB0(); \
      const bf16x4 h1 = RDW(BA, 1, 1, NKS, NCP); SB0(); \
      MF(acc[3][(CP) * 2 + 1], W[1], X[KS][3]); SB0(); H3; SB0(); \
      WN[0] = JOINW(l0, h0); WN[1] = JOINW(l1, h1); } while (0)
#define STEPL(X, KS, W, CP) do { _Pragma("unroll") for (int c = 0; c < 2; ++c) _Pragma("unroll") for (int i = 0; i < 4; ++i) MF(acc[i][(CP) * 2 + c], W[c], X[KS][i]); } while (0)
#define NEEDX(U) do { if (__builtin_expect(!((have >> (2 * (U) + (wid >> 1))) & 1u), 0)) { \
        SPIN_UNTIL(((have = (unsigned)__ballot(ld_sys32(myws + WS_AFL + (q * 2 + m) * 128 + (lane & 31) * 4) >= call)) >> (2 * (U) + (wid >> 1))) & 1u); \
        if (dead) have = ~0u; } } while (0)
#define NOP() do {} while (0)
#define VADD(D, S, K) asm volatile("v_add_u32 %0, %2, %1" : "=v"(D) : "v"(S), "s"(K))
#define UNIT(X, XO, BA, BN, T, L1, L0, N0, N4, LAST) do { \
      int d_; unsigned u_; TR(trc, (T)); \
      STEP(X, 0, W0, 0, W2, BA, 0, 2, if (L1) LX(XO, 1, 0, xn), if (L1) LX(XO, 1, 1, xn), d_ = (((T) + 1) & (NB - 1)) ? B_SLOT : -(NB - 1) * B_SLOT); \
      STEP(X, 0, W1, 1, W3, BA, 0, 3, if (L1) LX(XO, 1, 2, xn), if (L1) LX(XO, 1, 3, xn), u_ = (unsigned)(utab >> (4 * (((T) + 2) & 15))) & 15u); \
      STEP(X, 0, W2, 2, W0, BA, 1, 0, VADD(BN[0][0], BA[0][0], d_), VADD(BN[0][1], BA[0][1], d_), NOP()); \
      STEP(X, 0, W3, 3, W1, BA, 1, 1, VADD(BN[1][0], BA[1][0], d_), VADD(BN[1][1], BA[1][1], d_), WAITX(X, 1, N4)); \
      STEP(X, 1, W0, 0, W2, BA, 1, 2, if (L0) xn = xa + (size_t)u_ * 8192, if (L0) NEEDX(u_), NOP()); \
      STEP(X, 1, W1, 1, W3, BA, 1, 3, NOP(), NOP(), NOP()); \
      if (!(LAST)) { \
        TR(trc, 16 + (T)); asm volatile("s_barrier" ::: "memory"); TR(trc, 32 + (T)); SB0();        \
        STEP(X, 1, W2, 2, W0, BN, 0, 0, if (L0) LX(X, 0, 0, xn), if (L0) LX(X, 0, 1, xn), NOP()); \
        STEP(X, 1, W3, 3, W1, BN, 0, 1, if (L0) LX(X, 0, 2, xn), if (L0) LX(X, 0, 3, xn), WAITX(XO, 0, N0)); } \
      else { STEPL(X, 1, W2, 2); STEPL(X, 1, W3, 3); } } while (0)
    if (CPRIO) __builtin_amdgcn_s_setprio(CPRIO);
    unsigned trc = 0;
    bar();
    LOADW(W0, BA0, 0, 0); LOADW(W1, BA0, 0, 1);
    WAITX(X0, 0, 8);
    SB0();
    STAMP(2);
    for (int t = 0; t < NU - 2; t += 2) { UNIT(X0, X1, BA0, BA1, t, true, true, 8, 8, false); UNIT(X1, X0, BA1, BA0, t + 1, true, true, 8, 8, false); }
    UNIT(X0, X1, BA0, BA1, NU - 2, true, false, 4, 8, false);
    UNIT(X1, X0, BA1, BA0, NU - 1, false, false, 0, 0, true);
#undef LX
#undef XADDR
#undef NOP
#undef NEEDX
#undef VADD
#undef WAITX
#undef WAITX_
#undef LX_
#undef TM3
#undef LOADW
#undef RDW
#undef JOINW
#undef MF
#undef SB0
#undef STEP
#undef STEPL
#undef UNIT
    asm volatile("s_nop 15\n\ts_nop 15" ::: "memory");
    STAMP(3);
  } else {
    const int pw = wid - 4;
    const char* wb = (const char*)(p.w + (size_t)(q * 1024) * ND + ctile * 128 + pw * 32);
    const unsigned wvo = (unsigned)((lane >> 3) * ND * 4 + (lane & 7) * 16);
    u32x4 rb0[8], rb1[8];
    const int woff0 = 512 * pw + 64 * (lane >> 3) + 16 * (((lane & 7) >> 1) ^ (lane >> 5)) + 8 * (lane & 1);
    const int woff1 = 512 * pw + 64 * (lane >> 3) + 16 * (((lane & 7) >> 1) ^ (2 | (lane >> 5))) + 8 * (lane & 1);
    while (lds_get(miscA + 25 * 4) < 4) __builtin_amdgcn_s_sleep(1);
    issueB(rb0, wb, wvo, unit_of(0, m, n));
    unsigned trp = 0;
    WAIT_V(0); writeB(rb0, smem, woff0, woff1, 0);
    bar();
    issueB(rb1, wb, wvo, unit_of(1, m, n));
    issueB(rb0, wb, wvo, unit_of(2, m, n));
    WAIT_V(8); writeB(rb1, smem, woff0, woff1, 1);
    issueB(rb1, wb, wvo, unit_of(3, m, n));
    bar();
#define PSTEP(s, RB, MORE, NW) do { \
      TR(trp, (s)); WAIT_V(NW); TR(trp, 16 + (s)); \
      writeB(RB, smem, woff0, woff1, (s) & (NB - 1)); \
      if (MORE) { TR(trp, 32 + (s)); issueB(RB, wb, wvo, unit_of((s) + 2, m, n)); TR(trp, 48 + (s)); } \
      bar(); } while (0)
    for (int s = 2; s < NU - 2; s += 2) { PSTEP(s, rb0, true, 8); PSTEP(s + 1, rb1, true, 8); }
    PSTEP(NU - 2, rb0, false, 8); PSTEP(NU - 1, rb1, false, 0);
#undef PSTEP
  }
  bar();
  asm volatile("" : "+v"(tid));
  lane = tid & 63;

  const int tile = m * 32 + ctile;
  call = misc[0];
  const unsigned par = call & 1;
  const unsigned ldsb = miscA - LDS_MISC;
  if (wid < 4) {
#pragma unroll
    for (int i = 1; i < 4; ++i) {
      char* dst = myws + WS_XB + ((size_t)(tile * 4 + ((q + i) & 3)) * 3 + (i - 1)) * 16384 + wid * 4096 + lane * 16;
#pragma unroll
      for (int e = 0; e < 4; ++e) {
        u32x4 o = {cvt_pk(acc[i][2 * e][0], acc[i][2 * e][1]), cvt_pk(acc[i][2 * e][2], acc[i][2 * e][3]),
                   cvt_pk(acc[i][2 * e + 1][0], acc[i][2 * e + 1][1]), cvt_pk(acc[i][2 * e + 1][2], acc[i][2 * e + 1][3])};
        st_loc128(dst + e * 1024, o);
      }
    }
    unsigned sel_lo = 0x00003f80u, sel_hi = 0x3f800000u;
    asm volatile("" : "+v"(sel_lo), "+v"(sel_hi));
    u32x4 pr[3][4];
    char* src = myws + WS_XB + (size_t)(tile * 4 + q) * 3 * 16384 + wid * 4096 + lane * 16;
    auto fetch = [&]() {
#pragma unroll
      for (int i = 0; i < 3; ++i)
#pragma unroll
        for (int e = 0; e < 4; ++e) asm volatile("global_load_dwordx4 %0, %1, off sc0 sc1" : "=v"(pr[i][e]) : "v"(src + i * 16384 + e * 1024) : "memory");
      asm volatile("s_waitcnt vmcnt(0)" : "+v"(pr[0][0]), "+v"(pr[0][1]), "+v"(pr[0][2]), "+v"(pr[0][3]), "+v"(pr[1][0]), "+v"(pr[1][1]), "+v"(pr[1][2]), "+v"(pr[1][3]),
                   "+v"(pr[2][0]), "+v"(pr[2][1]), "+v"(pr[2][2]), "+v"(pr[2][3]) :: "memory");
      unsigned mn = 0xffffffffu;
#pragma unroll
      for (int i = 0; i < 3; ++i)
#pragma unroll
        for (int e = 0; e < 4; ++e) mn = min(min(mn, min(pr[i][e][0], pr[i][e][1])), min(pr[i][e][2], pr[i][e][3]));
      return __all(mn != 0u) != 0; };
    WAIT_V(0);
    if (lane < 3) st_sys32(myws + WS_XFL + (size_t)(tile * 4 + ((q + lane + 1) & 3)) * 128 + (lane * 4 + wid) * 4, call);
    SPIN_UNTIL(__all(ld_sys32(myws + WS_XFL + (size_t)(tile * 4 + q) * 128 + ((lane < 3 ? lane : 0) * 4 + wid) * 4) >= call));
    fetch();
    STAMP(4); RSTAMP(0);
#pragma unroll
    for (int i = 0; i < 3; ++i)
#pragma unroll
      for (int e = 0; e < 4; ++e)
#pragma unroll
        for (int d = 0; d < 4; ++d) {
          float a0 = acc[0][2 * e + (d >> 1)][(d & 1) * 2], a1 = acc[0][2 * e + (d >> 1)][(d & 1) * 2 + 1];
          asm volatile("v_dot2c_f32_bf16 %0, %1, %2" : "+v"(a0) : "v"(pr[i][e][d]), "v"(sel_lo));
          asm volatile("v_dot2c_f32_bf16 %0, %1, %2" : "+v"(a1) : "v"(pr[i][e][d]), "v"(sel_hi));
          acc[0][2 * e + (d >> 1)][(d & 1) * 2] = a0; acc[0][2 * e + (d >> 1)][(d & 1) * 2 + 1] = a1;
        }
    asm volatile("s_nop 3" ::: "memory");
    float mx = 0.f;
#pragma unroll
    for (int nb = 0; nb < 8; ++nb)
#pragma unroll
      for (int r = 0; r < 4; ++r) { const float v = fmaxf(acc[0][nb][r], 0.f); acc[0][nb][r] = v; mx = fmaxf(mx, v); }
    mx = __uint_as_float(wave_max_u32(__float_as_uint(mx) & 0x7fffffffu));
    if (lane == 0) {
      unsigned old;
      asm volatile("ds_max_u32 %1, %2\n\tds_add_rtn_u32 %0, %3, %4\n\ts_waitcnt lgkmcnt(0)" : "=v"(old) : "v"(miscA + 4 * 4), "v"(__float_as_uint(mx)), "v"(miscA + 5 * 4), "v"(1u) : "memory");
      if (old == 3) st_sys64(myws + WS_LMAX + par * 2048 + b * 8, ((unsigned long long)call << 32) | misc[4]);
    }
    STAMP(5);
    const float rinv = mx > 0.f ? __builtin_amdgcn_rcpf(mx) : 0.f;
    unsigned pk[16], mask = 0;
#pragma unroll
    for (int i = 0; i < 16; ++i) { const f32x2 t2 = (f32x2){acc[0][i >> 1][(i & 1) * 2], acc[0][i >> 1][(i & 1) * 2 + 1]} * rinv;
      pk[i] = __builtin_bit_cast(unsigned, __builtin_amdgcn_cvt_pknorm_u16(t2[0], t2[1])); }
#pragma unroll
    for (int i = 31; i >= 0; --i) mask = __builtin_amdgcn_alignbit(mask, 0u - __float_as_uint(acc[0][i >> 2][i & 3]), 31);
    const unsigned cnt = __builtin_popcount(mask);
    const unsigned incl = wave_scan_u32(cnt);
    const unsigned total = __builtin_amdgcn_readlane(incl, 63);
    const unsigned sb_ = ldsb + wid * CHUNK;
    *(LAS unsigned*)(size_t)(sb_ + 16 + lane * 4) = mask;
    if (lane == 0) *(LAS u32x4*)(size_t)sb_ = (u32x4){total, __float_as_uint(mx), dead ? 1u : 0u, 0u};
    unsigned off = sb_ + 272 + 2 * (incl - cnt);
#define PUTC(I, OP) asm volatile("v_cmpx_ne_u32 vcc, 0, %1\n\t" OP " %0, %2\n\tv_add_u32 %0, 2, %0\n\ts_mov_b64 exec, -1" : "+v"(off) : "v"(acc[0][(I) >> 2][(I) & 3]), "v"(pk[(I) >> 1]) : "vcc", "memory")
#pragma unroll
    for (int i = 0; i < 32; i += 2) { PUTC(i, "ds_write_b16"); PUTC(i + 1, "ds_write_b16_d16_hi"); }
#undef PUTC
    const int dstr = ctile >> 2, idx = ((ctile & 3) * 2 + m) * 4 + q;
    char* out_ = myws + WS_STG + ((size_t)((par * 8 + dstr) * 32 + idx) * 4 + wid) * CHUNK;
    const unsigned units = (272 + 2 * total + 15) >> 4;
    asm volatile("s_waitcnt lgkmcnt(0)" ::: "memory");
    STAMP(9);
    for (unsigned u = lane; u < ((units + 63) & ~63u); u += 64)
      if (u < units) st_wt128(out_ + u * 16, *(const LAS u32x4*)(size_t)(sb_ + u * 16));
    STAMP(10);
    WAIT_V(0);
    if (lane == 0) { st_sys64(p.ws[dstr] + WS_TFL + ((size_t)((par * 8 + p.rank) * 32 + idx) * 4 + wid) * 8, ((unsigned long long)call << 32) | units);
      asm volatile("ds_add_u32 %0, %1" ::"v"(miscA + 27 * 4), "v"(1u) : "memory"); }
    STAMP(7); RSTAMP(1);
  } else {
    const int pw = wid - 4, srcr = RSRC(b), idx = RIDX(b);
    const char* in_ = p.ws[srcr] + WS_STG + ((size_t)((par * 8 + p.rank) * 32 + idx) * 4 + pw) * CHUNK;
    const unsigned rb_ = ldsb + 32768 + pw * CHUNK;
    float val[32]; float cscale = 0.f;
    bool gdone = !(b == RGATH(p.rank) && wid == 7), tdone = false, adone = wid != 4;
    while (lds_get(miscA + 27 * 4) < 4) __builtin_amdgcn_s_sleep(4);
    RSTAMP(0);
    STAMP(11);
    bool seen = false;
    bool bad = false;
    auto progress = [&]() {
      unsigned long long gv[4], am = 0, tf = 0;
      unsigned tm = 0;
      const bool pollt = wid == 4 && !seen;
      if (!gdone) tm = ld_sys32(myws + WS_TMO);
      if (pollt) tf = ld_sys64(myws + WS_TFL + ((size_t)((par * 8 + srcr) * 32 + idx) * 4 + (lane & 3)) * 8);
      if (!adone) am = ld_sys64(myws + WS_GMAX + par * 1024 + (b & 7) * 128 + (lane & 15) * 8);
      if (!gdone) {
#pragma unroll
        for (int i = 0; i < 4; ++i) gv[i] = ld_sys64(myws + WS_LMAX + par * 2048 + (i * 64 + lane) * 8);
      }
      bad |= !gdone && tm == call;
      if (!gdone) {
        bool ok = true; unsigned lm = 0;
#pragma unroll
        for (int i = 0; i < 4; ++i) { ok &= (unsigned)(gv[i] >> 32) == call; lm = max(lm, (unsigned)gv[i]); }
        if (__all(ok)) {
          lm = wave_max_u32(lm);
          if (__builtin_amdgcn_readfirstlane(bad)) lm = 0xffffffffu;
          st_sys64(p.ws[lane >> 3] + WS_GMAX + par * 1024 + (lane & 7) * 128 + p.rank * 8, ((unsigned long long)call << 32) | lm);
          gdone = true;
        }
      }
      if (!adone && __all((lane & 8) || (unsigned)(am >> 32) == call)) {
        const bool tmo_ = (unsigned)__builtin_amdgcn_readlane((unsigned)am, 8) == call;
        lds_put(miscA + 1 * 4, wave_max_u32((lane & 8) ? 0u : (unsigned)am)); lds_put(miscA + 2 * 4, tmo_ ? 1u : 0u); lds_put(miscA + 26 * 4, 1); adone = true; STAMP(12);
      }
      if (pollt && __all((unsigned)(tf >> 32) == call)) {
        if (lane < 4) *(volatile LAS unsigned*)(size_t)(miscA + (32 + lane) * 4) = (unsigned)tf;
        lds_put(miscA + 28 * 4, 1);
      }
      if (!seen) seen = lds_get(miscA + 28 * 4) != 0;
      if (seen && !tdone && gdone) {
        {
          tf = lds_get(miscA + (32 + pw) * 4);
          STAMP(9); RSTAMP(1);
          const unsigned units = min((unsigned)__builtin_amdgcn_readfirstlane((unsigned)tf), (unsigned)(CHUNK / 16));
          u32x4 r[5];
#pragma unroll
          for (int k = 0; k < 5; ++k) { r[k] = (u32x4){0u, 0u, 0u, 0u}; if (k * 64 + lane < units) asm volatile("global_load_dwordx4 %0, %1, off sc0 sc1" : "+v"(r[k]) : "v"(in_ + (k * 64 + lane) * 16) : "memory"); }
          asm volatile("s_waitcnt vmcnt(0)" : "+v"(r[0]), "+v"(r[1]), "+v"(r[2]), "+v"(r[3]), "+v"(r[4]) :: "memory");
          u32x2 am2 = {0u, 0u};
          if (!adone) asm volatile("global_load_dwordx2 %0, %1, off sc0 sc1" : "+v"(am2) : "v"(myws + WS_GMAX + par * 1024 + (b & 7) * 128 + (lane & 15) * 8) : "memory");
          STAMP(10); RSTAMP(2);
#pragma unroll
          for (int k = 0; k < 5; ++k) if (k * 64 < units) *(LAS u32x4*)(size_t)(rb_ + min(k * 64 + lane, CHUNK / 16 - 1) * 16) = r[k];
          asm volatile("s_waitcnt lgkmcnt(0)" ::: "memory");
          const unsigned mask = *(const LAS unsigned*)(size_t)(rb_ + 16 + lane * 4);
          cscale = __uint_as_float(*(const LAS unsigned*)(size_t)(rb_ + 4));
          bad |= *(const LAS unsigned*)(size_t)(rb_ + 8) != 0u;
          const unsigned cnt = __builtin_popcount(mask);
          unsigned off = rb_ + 272 + 2 * (wave_scan_u32(cnt) - cnt);
          unsigned mm = __builtin_bitreverse32(mask);
          unsigned cv[32];
#pragma unroll
          for (int i = 0; i < 32; ++i) { cv[i] = 0;
            asm volatile("v_add_co_u32 %2, vcc, %2, %2\n\ts_nop 0\n\ts_mov_b64 exec, vcc\n\tds_read_u16 %0, %1\n\tv_add_u32 %1, 2, %1\n\ts_mov_b64 exec, -1" : "+v"(cv[i]), "+v"(off), "+v"(mm) :: "vcc", "memory"); }
          asm volatile("s_waitcnt lgkmcnt(0)" : "+v"(cv[0]), "+v"(cv[1]), "+v"(cv[2]), "+v"(cv[3]), "+v"(cv[4]), "+v"(cv[5]), "+v"(cv[6]), "+v"(cv[7]), "+v"(cv[8]), "+v"(cv[9]), "+v"(cv[10]), "+v"(cv[11]), "+v"(cv[12]), "+v"(cv[13]), "+v"(cv[14]), "+v"(cv[15]) :: "memory");
          asm volatile("" : "+v"(cv[16]), "+v"(cv[17]), "+v"(cv[18]), "+v"(cv[19]), "+v"(cv[20]), "+v"(cv[21]), "+v"(cv[22]), "+v"(cv[23]), "+v"(cv[24]), "+v"(cv[25]), "+v"(cv[26]), "+v"(cv[27]), "+v"(cv[28]), "+v"(cv[29]), "+v"(cv[30]), "+v"(cv[31]));
#pragma unroll
          for (int i = 0; i < 32; ++i) val[i] = (float)cv[i];
          asm volatile("s_waitcnt vmcnt(0)" : "+v"(am2) :: "memory");
          if (!adone && __all((lane & 8) || am2[1] == call)) {
            const bool tmo_ = (unsigned)__builtin_amdgcn_readlane(am2[0], 8) == call;
            lds_put(miscA + 1 * 4, wave_max_u32((lane & 8) ? 0u : am2[0])); lds_put(miscA + 2 * 4, tmo_ ? 1u : 0u); lds_put(miscA + 26 * 4, 1); adone = true; STAMP(12);
          }
          tdone = true;
        }
      }
      return gdone && tdone && adone; };
    SPIN_UNTIL(progress());
    STAMP(6);
    if (wid == 4) { if (!adone) { lds_put(miscA + 2 * 4, 1); lds_put(miscA + 26 * 4, 1); } }
    else { while (lds_get(miscA + 26 * 4) == 0) __builtin_amdgcn_s_sleep(1); }
    const unsigned amaxb = lds_get(miscA + 1 * 4);
    const float amax = __uint_as_float(amaxb);
    bad = __builtin_amdgcn_readfirstlane(bad) || dead || amaxb == 0xffffffffu || lds_get(miscA + 2 * 4) != 0;
    STAMP(7);
    const float cq = amax > 0.f ? cscale * (127.0f / 65535.0f) / amax : 0.f;
    const unsigned tb_ = ldsb + 65536 + pw * 2304;
#pragma unroll
    for (int nb = 0; nb < 8; ++nb) {
      unsigned v = 0;
#pragma unroll
      for (int e = 0; e < 4; ++e) v = __builtin_amdgcn_cvt_pk_u8_f32(fminf(val[nb * 4 + e] * cq, 127.0f), e, v);
      *(LAS unsigned*)(size_t)(tb_ + (lane & 15) * 144 + nb * 16 + (lane >> 4) * 4) = bad ? 0x80808080u : v;
    }
    asm volatile("s_waitcnt lgkmcnt(0)" ::: "memory");
    signed char* ob = p.out[p.rank] + (size_t)(srcr * MS + ((idx >> 2) & 1) * 256 + pw * 64 + (idx & 3) * 16) * NS + (idx >> 3) * 128;
#pragma unroll
    for (int k = 0; k < 2; ++k) { const int r = k * 8 + (lane >> 3);
      *(u32x4*)(ob + (size_t)r * NS + (lane & 7) * 16) = *(const LAS u32x4*)(size_t)(tb_ + r * 144 + (lane & 7) * 16); }
    RSTAMP(3);
  }
  STAMP(8);
}

static unsigned long long* g_stamps = nullptr;

extern "C" void kernel_launch(int rank, int nranks, ncclComm_t comm, hipStream_t stream,
                              void* const* d_in, const int* in_sizes, int n_in,
                              void* d_out, int out_size, const int* in_dtypes, int out_dtype,
                              void* const* d_out_all, void* const* d_ws_all, size_t ws_bytes) {
  if (nranks != NR || ws_bytes < WS_END || n_in < 2) { fprintf(stderr, "bad config\n"); return; }
  if (in_sizes[0] != MS * KD || in_sizes[1] != KD * ND || out_size != 4096 * NS) { fprintf(stderr, "bad sizes\n"); return; }
  HIP_CHECK(hipFuncSetAttribute((const void*)gemm_a2a, hipFuncAttributeMaxDynamicSharedMemorySize, LDS_BYTES));
  Params p;
  p.x = (const float*)d_in[0]; p.w = (const float*)d_in[1];
  for (int i = 0; i < NR; ++i) { p.ws[i] = (char*)d_ws_all[i]; p.out[i] = (signed char*)d_out_all[i]; }
  p.stamps = g_stamps;
  p.rank = rank; p.pad = 0;
  hipLaunchKernelGGL(gemm_a2a, dim3(NWG), dim3(NTHR), LDS_BYTES, stream, p);
  HIP_CHECK(hipGetLastError());
}
```

```cpp
#include <hip/hip_runtime.h>
#include <rccl/rccl.h>
#include <cstdio>
#include <cstdint>

#define HIP_CHECK(expr) do { hipError_t _e = (expr); if (_e != hipSuccess) { \
    fprintf(stderr, "HIP error %s at %s:%d\n", hipGetErrorString(_e), __FILE__, __LINE__); \
    return; } } while (0)

typedef unsigned u32x4 __attribute__((ext_vector_type(4)));
typedef unsigned u32x2 __attribute__((ext_vector_type(2)));
typedef float f32x4 __attribute__((ext_vector_type(4)));
typedef float f32x2 __attribute__((ext_vector_type(2)));
typedef short bf16x8 __attribute__((ext_vector_type(8)));
typedef short bf16x4 __attribute__((ext_vector_type(4)));
typedef __bf16 bf16x2_t __attribute__((ext_vector_type(2)));
#define LAS __attribute__((address_space(3)))

constexpr int NR = 8;
constexpr int MS = 512;
constexpr int KD = 4096, ND = 4096;
constexpr int NS = 512;
constexpr int NWG = 256, NTHR = 512;
constexpr int NU = 16;
constexpr unsigned long long DEADLINE = 40000000ull;

constexpr int B_SLOT = 16384, NB = 8;
constexpr int LDS_MISC = NB * B_SLOT, LDS_BYTES = LDS_MISC + 256;

constexpr size_t WS_CNT  = 0;
constexpr size_t WS_AFL  = WS_CNT + 256 * 128;
constexpr size_t WS_XFL  = WS_AFL + 8 * 128;
constexpr size_t WS_LMAX = WS_XFL + 256 * 128;
constexpr size_t WS_GMAX = WS_LMAX + 2 * 256 * 8;
constexpr size_t WS_TFL  = WS_GMAX + 2 * 8 * 128;
constexpr size_t WS_TMO  = WS_TFL + 16384;
constexpr size_t WS_XA   = 1 << 20;
constexpr size_t WS_XB   = WS_XA + (4 << 20);
constexpr int CHUNK = 4608;
constexpr size_t WS_STG  = WS_XB + (size_t)64 * 4 * 3 * 16384;
constexpr size_t WS_END  = WS_STG + (size_t)2 * 8 * 32 * 4 * CHUNK;
static_assert(WS_TMO + 128 <= WS_XA, "control area");

struct Params {
  const float* x; const float* w;
  char* myws; signed char* myout;
  int rank; int pad;
  char* ws[NR];
};

__device__ __forceinline__ unsigned cvt_pk(float lo, float hi) {
  f32x2 v = {lo, hi};
  return __builtin_bit_cast(unsigned, __builtin_convertvector(v, bf16x2_t));
}
__device__ __forceinline__ unsigned long long ld_sys64(const void* p) { return __hip_atomic_load((const unsigned long long*)p, __ATOMIC_RELAXED, __HIP_MEMORY_SCOPE_SYSTEM); }
__device__ __forceinline__ unsigned ld_sys32(const void* p) { return __hip_atomic_load((const unsigned*)p, __ATOMIC_RELAXED, __HIP_MEMORY_SCOPE_SYSTEM); }
__device__ __forceinline__ void st_sys64(void* p, unsigned long long v) { __hip_atomic_store((unsigned long long*)p, v, __ATOMIC_RELAXED, __HIP_MEMORY_SCOPE_SYSTEM); }
__device__ __forceinline__ void st_sys32(void* p, unsigned v) { __hip_atomic_store((unsigned*)p, v, __ATOMIC_RELAXED, __HIP_MEMORY_SCOPE_SYSTEM); }
__device__ __forceinline__ void st_wt128(void* p, u32x4 v) { asm volatile("global_store_dwordx4 %0, %1, off sc0 sc1\n\ts_nop 1" ::"v"(p), "v"(v) : "memory"); }
#define WAIT_V(n) asm volatile("s_waitcnt vmcnt(%0)" ::"n"(n) : "memory")
__device__ __forceinline__ void bar() { asm volatile("s_waitcnt lgkmcnt(0)\n\ts_barrier" ::: "memory"); }
__device__ __forceinline__ void lds_put(unsigned addr, unsigned v) { asm volatile("ds_write_b32 %0, %1" ::"v"(addr), "v"(v) : "memory"); }
__device__ __forceinline__ unsigned lds_get(unsigned addr) { unsigned r; asm volatile("ds_read_b32 %0, %1\n\ts_waitcnt lgkmcnt(0)" : "=v"(r) : "v"(addr) : "memory"); return __builtin_amdgcn_readfirstlane(r); }
__device__ __forceinline__ unsigned lds_min4(unsigned addr) {
  u32x4 r; asm volatile("ds_read_b128 %0, %1\n\ts_waitcnt lgkmcnt(0)" : "=v"(r) : "v"(addr) : "memory");
  return __builtin_amdgcn_readfirstlane(min(min(r.x, r.y), min(r.z, r.w)));
}
#define SPIN_UNTIL(COND) do { const unsigned long long t0_ = __builtin_amdgcn_s_memrealtime(); unsigned it_ = 0; \
    while (!(COND)) { \
      if ((++it_ & 63u) == 0u && (ld_sys32(myws + WS_TMO) == call || __builtin_amdgcn_s_memrealtime() - t0_ > DEADLINE)) { dead = true; break; } \
      __builtin_amdgcn_s_sleep(1); } \
    if (dead) { if (lane == 0) st_sys32(myws + WS_TMO, call); if (lane < 16) st_sys32(myws + WS_GMAX + lane * 128 + 64, call); } } while (0)

__device__ __forceinline__ int unit_of(int t, int m, int n) {
  int v = t;
  if (m) v = (t == 0) ? 1 : (t == 15) ? 14 : ((t & 1) ? t + 2 : t - 2);
  return (v + n) & 15;
}

__device__ __forceinline__ void issueB(u32x4 (&rb)[8], const char* wbase, unsigned voff, int unit) {
  const char* src = wbase + (size_t)unit * 64 * ND * 4;
#pragma unroll
  for (int i = 0; i < 8; ++i) asm volatile("global_load_dwordx4 %0, %1, %2" : "=v"(rb[i]) : "v"(voff), "s"(src + (size_t)i * 8 * ND * 4) : "memory");
}
__device__ __forceinline__ void writeB(u32x4 (&rb)[8], char* smem, int woff0, int woff1, int slot) {
  LAS char* B = (LAS char*)smem + slot * B_SLOT;
#pragma unroll
  for (int i = 0; i < 8; ++i) {
    asm volatile("" : "+v"(rb[i]));
    const f32x4 v = __builtin_bit_cast(f32x4, rb[i]);
    *(LAS u32x2*)(B + ((i & 1) ? woff1 : woff0) + i * 2048) = (u32x2){cvt_pk(v[0], v[1]), cvt_pk(v[2], v[3])};
  }
}

__device__ __forceinline__ unsigned wave_max_u32(unsigned v) {
  v = max(v, (unsigned)__builtin_amdgcn_update_dpp(0u, v, 0x111, 0xf, 0xf, false));
  v = max(v, (unsigned)__builtin_amdgcn_update_dpp(0u, v, 0x112, 0xf, 0xf, false));
  v = max(v, (unsigned)__builtin_amdgcn_update_dpp(0u, v, 0x114, 0xf, 0xf, false));
  v = max(v, (unsigned)__builtin_amdgcn_update_dpp(0u, v, 0x118, 0xf, 0xf, false));
  v = max(v, (unsigned)__builtin_amdgcn_update_dpp(0u, v, 0x142, 0xa, 0xf, false));
  v = max(v, (unsigned)__builtin_amdgcn_update_dpp(0u, v, 0x143, 0xc, 0xf, false));
  return __builtin_amdgcn_readlane(v, 63);
}
__device__ __forceinline__ unsigned wave_scan_u32(unsigned v) {
  v += __builtin_amdgcn_update_dpp(0u, v, 0x111, 0xf, 0xf, false);
  v += __builtin_amdgcn_update_dpp(0u, v, 0x112, 0xf, 0xf, false);
  v += __builtin_amdgcn_update_dpp(0u, v, 0x114, 0xf, 0xf, false);
  v += __builtin_amdgcn_update_dpp(0u, v, 0x118, 0xf, 0xf, false);
  v += __builtin_amdgcn_update_dpp(0u, v, 0x142, 0xa, 0xf, false);
  v += __builtin_amdgcn_update_dpp(0u, v, 0x143, 0xc, 0xf, false);
  return v;
}

__global__ __launch_bounds__(NTHR, 2) void gemm_a2a(Params p) {
  extern __shared__ __attribute__((aligned(16))) char smem[];
  int tid = threadIdx.x, lane = tid & 63;
  const int wid = __builtin_amdgcn_readfirstlane(tid >> 6);
  const int b = blockIdx.x;
  const int g = b & 7, q = g & 3, h = g >> 2, j = b >> 3, m = j & 1, n = j >> 1;
  const int ctile = h * 16 + n;
  char* myws = p.myws;
  volatile LAS unsigned* misc = (volatile LAS unsigned*)(smem + LDS_MISC);
  const unsigned miscA = (unsigned)(size_t)(LAS char*)smem + LDS_MISC;
  if (tid < 64) misc[tid] = 0;
  if (wid >= 4) bar();

  f32x4 acc[4][8];
  unsigned call = 0;
  bool dead = false;

  if (wid < 4) {
    u32x4 X0[2][4], X1[2][4];
    unsigned long long utab;
    unsigned BA0[2][2], BA1[2][2];
    unsigned have = 0;
    unsigned tw[16];
    const int pu = unit_of(0, m, n), pc = pu * 2 + h;
    {
      const int iw = (pc & 1) * 2 + (wid >> 1), ks = wid & 1;
      u32x4 xr[8];
      const float* xb = p.x + (size_t)(m * 256 + iw * 64 + (lane >> 3)) * KD + q * 1024 + pu * 64 + ks * 32 + (lane & 7) * 4;
#pragma unroll
      for (int i = 0; i < 8; ++i) {
        const float* ptr = xb + (size_t)(((((i >> 1) + q) & 3) * 16) + (i & 1) * 8) * KD;
        asm volatile("global_load_dwordx4 %0, %1, off" : "=v"(xr[i]) : "v"(ptr) : "memory");
      }
      bar();
      if (tid == 0) { call = ld_sys32(myws + WS_CNT + b * 128) + 1; }
    if (wid == 3) {
      const int tile_ = m * 32 + ctile, dstr_ = ctile >> 2;
      const char* a_[16];
      a_[0] = myws + WS_XA + (size_t)(q * 2 + m) * 524288;  a_[1] = a_[0] + 262144;  a_[2] = a_[0] + 524288 - 128;
#pragma unroll
      for (int i = 0; i < 4; ++i) a_[3 + i] = myws + WS_XB + (size_t)(tile_ * 4 + i) * 3 * 16384 + 16384;
      { const int idx_ = ((ctile & 3) * 2 + m) * 4 + q;
        a_[7] = myws + WS_STG + (size_t)(dstr_ * 32 + idx_) * 4 * CHUNK;          a_[8] = a_[7] + (size_t)8 * 32 * 4 * CHUNK;
        a_[9] = p.ws[(b & 7)] + WS_STG + (size_t)(p.rank * 32 + (b >> 3)) * 4 * CHUNK; a_[10] = a_[9] + (size_t)8 * 32 * 4 * CHUNK; }
      a_[11] = (const char*)p.myout + (size_t)((b & 7) * MS + (((b >> 3) >> 2) & 1) * 256 + ((b >> 3) & 3) * 16) * NS;
      const int pr_ = (b >> 3) & 7;
      a_[12] = p.ws[pr_] + WS_GMAX;
      a_[13] = p.ws[dstr_] + WS_TFL; a_[14] = myws + WS_XFL + (size_t)tile_ * 512; a_[15] = myws + WS_LMAX;
#pragma unroll
      for (int i = 0; i < 16; ++i) asm volatile("s_load_dword %0, %1, 0x0" : "=s"(tw[i]) : "s"(a_[i]) : "memory");
    }
      WAIT_V(0);
#pragma unroll
      for (int i = 0; i < 8; ++i) asm volatile("" : "+v"(xr[i]));
      char* xa = myws + WS_XA + ((size_t)(((q * 2 + m) * 4 + iw) * 16 + pu) * 8 + ks * 4) * 1024;
      const bool odd = lane & 1;
      const int dl = (lane >> 3) + (odd ? 8 : 0) + 16 * ((lane & 7) >> 1);
#pragma unroll
      for (int i = 0; i < 8; i += 2) {
        const f32x4 a = __builtin_bit_cast(f32x4, xr[i]), c = __builtin_bit_cast(f32x4, xr[i + 1]);
        const unsigned a0 = cvt_pk(a[0], a[1]), a1 = cvt_pk(a[2], a[3]), c0 = cvt_pk(c[0], c[1]), c1 = cvt_pk(c[2], c[3]);
        const unsigned s0 = odd ? a0 : c0, s1 = odd ? a1 : c1;
        const unsigned r0 = __builtin_amdgcn_mov_dpp(s0, 0xB1, 0xF, 0xF, true), r1 = __builtin_amdgcn_mov_dpp(s1, 0xB1, 0xF, 0xF, true);
        const u32x4 o = odd ? (u32x4){r0, r1, c0, c1} : (u32x4){a0, a1, r0, r1};
        st_wt128(xa + (i >> 1) * 1024 + dl * 16, o);
      }
#pragma unroll
      for (int i = 0; i < 4; ++i)
#pragma unroll
        for (int nb = 0; nb < 8; ++nb) { acc[i][nb] = (f32x4){0.f, 0.f, 0.f, 0.f}; asm volatile("" : "+v"(acc[i][nb])); }
#pragma unroll
      for (int ks = 0; ks < 2; ++ks)
#pragma unroll
        for (int i = 0; i < 4; ++i) { X0[ks][i] = (u32x4){0, 0, 0, 0}; X1[ks][i] = (u32x4){0, 0, 0, 0}; asm volatile("" : "+v"(X0[ks][i]), "+v"(X1[ks][i])); }
    utab = 0;
#pragma unroll
    for (int t = 0; t < NU; ++t) utab |= (unsigned long long)unit_of(t, m, n) << (4 * t);
      asm volatile("" : "+s"(utab));
    {
      const int gg = lane >> 4, q4 = (lane & 15) >> 2, pp = lane & 3;
#pragma unroll
      for (int t = 0; t < 2; ++t)
#pragma unroll
        for (int cp = 0; cp < 2; ++cp) {
          BA0[t][cp] = (miscA - LDS_MISC) + 2048 * gg + 64 * (4 * t + q4) + 16 * ((2 * cp + (pp >> 1)) ^ ((2 * gg + t) & 3)) + 8 * (pp & 1);
          BA1[t][cp] = BA0[t][cp]; asm volatile("" : "+v"(BA0[t][cp]), "+v"(BA1[t][cp]));
        }
    }
      WAIT_V(0);
      call = __builtin_amdgcn_readfirstlane(call);
      lds_put(miscA + (20 + wid) * 4, 1);
      if (wid == 0) {
        while (lds_min4(miscA + 20 * 4) == 0) __builtin_amdgcn_s_sleep(1);

        if (lane == 0) { st_sys32(myws + WS_AFL + (q * 2 + m) * 128 + pc * 4, call); st_sys32(myws + WS_CNT + b * 128, call); misc[0] = call; misc[16] = call; }
        lds_put(miscA + 17 * 4, 1);
        const unsigned need = 3u << (2 * unit_of(0, m, n));
        SPIN_UNTIL(((have = (unsigned)__ballot(ld_sys32(myws + WS_AFL + (q * 2 + m) * 128 + (lane & 31) * 4) >= call)) & need) == need);
        if (dead) have = ~0u;
        lds_put(miscA + 19 * 4, have);
        lds_put(miscA + 18 * 4, 1);
      } else {
        if (wid == 3) asm volatile("s_waitcnt lgkmcnt(0)" : "+s"(tw[0]), "+s"(tw[1]), "+s"(tw[2]), "+s"(tw[3]), "+s"(tw[4]), "+s"(tw[5]), "+s"(tw[6]), "+s"(tw[7]),
                                   "+s"(tw[8]), "+s"(tw[9]), "+s"(tw[10]), "+s"(tw[11]), "+s"(tw[12]), "+s"(tw[13]), "+s"(tw[14]), "+s"(tw[15]) :: "memory");
        u32x4 w4;
        for (;;) { asm volatile("ds_read_b128 %0, %1\n\ts_waitcnt lgkmcnt(0)" : "=v"(w4) : "v"(miscA + 16 * 4) : "memory");
          if (__builtin_amdgcn_readfirstlane(w4[2]) != 0) break;
          __builtin_amdgcn_s_sleep(1); }
        have = __builtin_amdgcn_readfirstlane(w4[3]);
        call = __builtin_amdgcn_readfirstlane(w4[0]);
      }
    }
    const char* xa = myws + WS_XA + (size_t)((q * 2 + m) * 4 + wid) * 16 * 8192;
    const unsigned xv0 = lane * 16, xv1 = xv0 + 4096;
#define XADDR(T) (xa + (size_t)((unsigned)(utab >> (4 * (T))) & 15u) * 8192)
#define LX(X, KS, I, A) asm volatile("global_load_dwordx4 %0, %1, %2 offset:%3 sc0 sc1" : "+v"(X[KS][I]) : "v"((KS) ? xv1 : xv0), "s"(A), "n"((I) * 1024) : "memory")
#define WAITX(X, KS, N) asm volatile("s_waitcnt vmcnt(%4)" : "+v"(X[KS][0]), "+v"(X[KS][1]), "+v"(X[KS][2]), "+v"(X[KS][3]) : "n"(N) : "memory")
#define NEEDX(U) do { if (__builtin_expect(!((have >> (2 * (U) + (wid >> 1))) & 1u), 0)) { \
        SPIN_UNTIL(((have = (unsigned)__ballot(ld_sys32(myws + WS_AFL + (q * 2 + m) * 128 + (lane & 31) * 4) >= call)) >> (2 * (U) + (wid >> 1))) & 1u); \
        if (dead) have = ~0u; } } while (0)
    const char* xn = XADDR(0);
    LX(X0, 0, 0, xn); LX(X0, 0, 1, xn); LX(X0, 0, 2, xn); LX(X0, 0, 3, xn);
    LX(X0, 1, 0, xn); LX(X0, 1, 1, xn); LX(X0, 1, 2, xn); LX(X0, 1, 3, xn);
    xn = XADDR(1);
    NEEDX((unsigned)unit_of(1, m, n));
    LX(X1, 0, 0, xn); LX(X1, 0, 1, xn); LX(X1, 0, 2, xn); LX(X1, 0, 3, xn);
    bf16x8 W0[2], W1[2], W2[2], W3[2];
#define RDW(BA, C, T, KS, CP) __builtin_amdgcn_ds_read_tr16_b64_v4i16((LAS bf16x4*)(size_t)(BA[T][C] + (KS) * 8192 + (CP) * 512))
#define JOINW(lo, hi) ((bf16x8){lo[0], lo[1], lo[2], lo[3], hi[0], hi[1], hi[2], hi[3]})
#define MF(ACC, W, X) asm volatile("v_mfma_f32_16x16x32_bf16 %0, %1, %2, %0" : "+v"(ACC) : "v"(W), "v"(X))
#define SB0() __builtin_amdgcn_sched_barrier(0)
#define LOADW(W, BA, KS, CP) do { \
      { const bf16x4 lo = RDW(BA, 0, 0, KS, CP), hi = RDW(BA, 0, 1, KS, CP); W[0] = JOINW(lo, hi); } \
      { const bf16x4 lo = RDW(BA, 1, 0, KS, CP), hi = RDW(BA, 1, 1, KS, CP); W[1] = JOINW(lo, hi); } } while (0)
#define STEP(X, KS, W, CP, WN, BA, NKS, NCP, H1, H2, H3) do { \
      MF(acc[0][(CP) * 2], W[0], X[KS][0]); SB0(); \
      const bf16x4 l0 = RDW(BA, 0, 0, NKS, NCP); SB0(); \
      MF(acc[1][(CP) * 2], W[0], X[KS][1]); SB0(); H1; SB0(); MF(acc[2][(CP) * 2], W[0], X[KS][2]); SB0(); \
      const bf16x4 h0 = RDW(BA, 0, 1, NKS, NCP); SB0(); \
      MF(acc[3][(CP) * 2], W[0], X[KS][3]); MF(acc[0][(CP) * 2 + 1], W[1], X[KS][0]); SB0(); \
      const bf16x4 l1 = RDW(BA, 1, 0, NKS, NCP); SB0(); \
      MF(acc[1][(CP) * 2 + 1], W[1], X[KS][1]); SB0(); H2; SB0(); MF(acc[2][(CP) * 2 + 1], W[1], X[KS][2]); SB0(); \
      const bf16x4 h1 = RDW(BA, 1, 1, NKS, NCP); SB0(); \
      MF(acc[3][(CP) * 2 + 1], W[1], X[KS][3]); SB0(); H3; SB0(); \
      WN[0] = JOINW(l0, h0); WN[1] = JOINW(l1, h1); } while (0)
#define STEPL(X, KS, W, CP) do { _Pragma("unroll") for (int c = 0; c < 2; ++c) _Pragma("unroll") for (int i = 0; i < 4; ++i) MF(acc[i][(CP) * 2 + c], W[c], X[KS][i]); } while (0)
#define NOP() do {} while (0)
#define VADD(D, S, K) asm volatile("v_add_u32 %0, %2, %1" : "=v"(D) : "v"(S), "s"(K))
#define UNIT(X, XO, BA, BN, T, L1, L0, N0, N4, LAST) do { \
      int d_; unsigned u_; \
      STEP(X, 0, W0, 0, W2, BA, 0, 2, if (L1) LX(XO, 1, 0, xn), if (L1) LX(XO, 1, 1, xn), d_ = (((T) + 1) & (NB - 1)) ? B_SLOT : -(NB - 1) * B_SLOT); \
      STEP(X, 0, W1, 1, W3, BA, 0, 3, if (L1) LX(XO, 1, 2, xn), if (L1) LX(XO, 1, 3, xn), u_ = (unsigned)(utab >> (4 * (((T) + 2) & 15))) & 15u); \
      STEP(X, 0, W2, 2, W0, BA, 1, 0, VADD(BN[0][0], BA[0][0], d_), VADD(BN[0][1], BA[0][1], d_), NOP()); \
      STEP(X, 0, W3, 3, W1, BA, 1, 1, VADD(BN[1][0], BA[1][0], d_), VADD(BN[1][1], BA[1][1], d_), WAITX(X, 1, N4)); \
      STEP(X, 1, W0, 0, W2, BA, 1, 2, if (L0) xn = xa + (size_t)u_ * 8192, if (L0) NEEDX(u_), NOP()); \
      STEP(X, 1, W1, 1, W3, BA, 1, 3, NOP(), NOP(), NOP()); \
      if (!(LAST)) { \
        asm volatile("s_barrier" ::: "memory"); SB0();        \
        STEP(X, 1, W2, 2, W0, BN, 0, 0, if (L0) LX(X, 0, 0, xn), if (L0) LX(X, 0, 1, xn), NOP()); \
        STEP(X, 1, W3, 3, W1, BN, 0, 1, if (L0) LX(X, 0, 2, xn), if (L0) LX(X, 0, 3, xn), WAITX(XO, 0, N0)); } \
      else { STEPL(X, 1, W2, 2); STEPL(X, 1, W3, 3); } } while (0)
    bar();
    LOADW(W0, BA0, 0, 0); LOADW(W1, BA0, 0, 1);
    WAITX(X0, 0, 8);
    SB0();

    for (int t = 0; t < NU - 2; t += 2) { UNIT(X0, X1, BA0, BA1, t, true, true, 8, 8, false); UNIT(X1, X0, BA1, BA0, t + 1, true, true, 8, 8, false); }
    UNIT(X0, X1, BA0, BA1, NU - 2, true, false, 4, 8, false);
    UNIT(X1, X0, BA1, BA0, NU - 1, false, false, 0, 0, true);
#undef LX
#undef XADDR
#undef NOP
#undef NEEDX
#undef VADD
#undef WAITX
#undef LOADW
#undef RDW
#undef JOINW
#undef MF
#undef SB0
#undef STEP
#undef STEPL
#undef UNIT
    asm volatile("s_nop 15\n\ts_nop 15" ::: "memory");
  } else {
    const int pw = wid - 4;
    const char* wb = (const char*)(p.w + (size_t)(q * 1024) * ND + ctile * 128 + pw * 32);
    const unsigned wvo = (unsigned)((lane >> 3) * ND * 4 + (lane & 7) * 16);
    u32x4 rb0[8], rb1[8];
    const int woff0 = 512 * pw + 64 * (lane >> 3) + 16 * (((lane & 7) >> 1) ^ (lane >> 5)) + 8 * (lane & 1);
    const int woff1 = 512 * pw + 64 * (lane >> 3) + 16 * (((lane & 7) >> 1) ^ (2 | (lane >> 5))) + 8 * (lane & 1);
    while (lds_get(miscA + 17 * 4) == 0) __builtin_amdgcn_s_sleep(1);
    issueB(rb0, wb, wvo, unit_of(0, m, n));
    WAIT_V(0); writeB(rb0, smem, woff0, woff1, 0);
    bar();
    issueB(rb1, wb, wvo, unit_of(1, m, n));
    issueB(rb0, wb, wvo, unit_of(2, m, n));
    WAIT_V(8); writeB(rb1, smem, woff0, woff1, 1);
    issueB(rb1, wb, wvo, unit_of(3, m, n));
    bar();
#define PSTEP(s, RB, MORE, NW) do { \
      WAIT_V(NW); \
      writeB(RB, smem, woff0, woff1, (s) & (NB - 1)); \
      if (MORE) { issueB(RB, wb, wvo, unit_of((s) + 2, m, n)); } \
      bar(); } while (0)
    for (int s = 2; s < NU - 2; s += 2) { PSTEP(s, rb0, true, 8); PSTEP(s + 1, rb1, true, 8); }
    PSTEP(NU - 2, rb0, false, 8); PSTEP(NU - 1, rb1, false, 0);
#undef PSTEP
  }
  asm volatile("" : "+v"(tid));
  lane = tid & 63;

  const int tile = m * 32 + ctile;
  if (wid >= 4) call = lds_get(miscA);
  const unsigned par = call & 1;
  const unsigned ldsb = miscA - LDS_MISC;
  if (wid < 4) {
#pragma unroll
    for (int i = 1; i < 4; ++i) {
      char* dst = myws + WS_XB + ((size_t)(tile * 4 + ((q + i) & 3)) * 3 + (i - 1)) * 16384 + wid * 4096 + lane * 16;
#pragma unroll
      for (int e = 0; e < 4; ++e) {
        u32x4 o = {cvt_pk(acc[i][2 * e][0], acc[i][2 * e][1]), cvt_pk(acc[i][2 * e][2], acc[i][2 * e][3]),
                   cvt_pk(acc[i][2 * e + 1][0], acc[i][2 * e + 1][1]), cvt_pk(acc[i][2 * e + 1][2], acc[i][2 * e + 1][3])};
        st_wt128(dst + e * 1024, o);
      }
    }
    unsigned sel_lo = 0x00003f80u, sel_hi = 0x3f800000u;
    asm volatile("" : "+v"(sel_lo), "+v"(sel_hi));
    u32x4 pr[3][4];
    char* src = myws + WS_XB + (size_t)(tile * 4 + q) * 3 * 16384 + wid * 4096 + lane * 16;
    WAIT_V(0);
    if (lane < 3) st_sys32(myws + WS_XFL + (size_t)(tile * 4 + ((q + lane + 1) & 3)) * 128 + (lane * 4 + wid) * 4, call);
    SPIN_UNTIL(__all(ld_sys32(myws + WS_XFL + (size_t)(tile * 4 + q) * 128 + ((lane < 3 ? lane : 0) * 4 + wid) * 4) >= call));
#pragma unroll
    for (int i = 0; i < 3; ++i)
#pragma unroll
      for (int e = 0; e < 4; ++e) asm volatile("global_load_dwordx4 %0, %1, off sc0 sc1" : "=v"(pr[i][e]) : "v"(src + i * 16384 + e * 1024) : "memory");
    asm volatile("s_waitcnt vmcnt(0)" : "+v"(pr[0][0]), "+v"(pr[0][1]), "+v"(pr[0][2]), "+v"(pr[0][3]), "+v"(pr[1][0]), "+v"(pr[1][1]), "+v"(pr[1][2]), "+v"(pr[1][3]),
                 "+v"(pr[2][0]), "+v"(pr[2][1]), "+v"(pr[2][2]), "+v"(pr[2][3]) :: "memory");

#pragma unroll
    for (int i = 0; i < 3; ++i)
#pragma unroll
      for (int e = 0; e < 4; ++e)
#pragma unroll
        for (int d = 0; d < 4; ++d) {
          float a0 = acc[0][2 * e + (d >> 1)][(d & 1) * 2], a1 = acc[0][2 * e + (d >> 1)][(d & 1) * 2 + 1];
          asm volatile("v_dot2c_f32_bf16 %0, %1, %2" : "+v"(a0) : "v"(pr[i][e][d]), "v"(sel_lo));
          asm volatile("v_dot2c_f32_bf16 %0, %1, %2" : "+v"(a1) : "v"(pr[i][e][d]), "v"(sel_hi));
          acc[0][2 * e + (d >> 1)][(d & 1) * 2] = a0; acc[0][2 * e + (d >> 1)][(d & 1) * 2 + 1] = a1;
        }
    asm volatile("s_nop 3" ::: "memory");
    float mx = 0.f;
#pragma unroll
    for (int nb = 0; nb < 8; ++nb)
#pragma unroll
      for (int r = 0; r < 4; ++r) { const float v = fmaxf(acc[0][nb][r], 0.f); acc[0][nb][r] = v; mx = fmaxf(mx, v); }
    mx = __uint_as_float(wave_max_u32(__float_as_uint(mx) & 0x7fffffffu));
    const float rinv = mx > 0.f ? __builtin_amdgcn_rcpf(mx) : 0.f;
    unsigned pk[16], mask = 0;
#pragma unroll
    for (int i = 0; i < 16; ++i) { const f32x2 t2 = (f32x2){acc[0][i >> 1][(i & 1) * 2], acc[0][i >> 1][(i & 1) * 2 + 1]} * rinv;
      pk[i] = __builtin_bit_cast(unsigned, __builtin_amdgcn_cvt_pknorm_u16(t2[0], t2[1])); }
#pragma unroll
    for (int i = 31; i >= 0; --i) mask = __builtin_amdgcn_alignbit(mask, 0u - __float_as_uint(acc[0][i >> 2][i & 3]), 31);
    const unsigned cnt = __builtin_popcount(mask);
    const unsigned incl = wave_scan_u32(cnt);
    const unsigned total = __builtin_amdgcn_readlane(incl, 63);
    const unsigned sb_ = ldsb + wid * CHUNK;
    *(LAS unsigned*)(size_t)(sb_ + 16 + lane * 4) = mask;
    if (lane == 0) *(LAS u32x4*)(size_t)sb_ = (u32x4){total, __float_as_uint(mx), dead ? 1u : 0u, 0u};
    unsigned off = sb_ + 272 + 2 * (incl - cnt);
#define PUTC(I, OP) asm volatile("v_cmpx_ne_u32 vcc, 0, %1\n\t" OP " %0, %2\n\tv_add_u32 %0, 2, %0\n\ts_mov_b64 exec, -1" : "+v"(off) : "v"(acc[0][(I) >> 2][(I) & 3]), "v"(pk[(I) >> 1]) : "vcc", "memory")
#pragma unroll
    for (int i = 0; i < 32; i += 2) { PUTC(i, "ds_write_b16"); PUTC(i + 1, "ds_write_b16_d16_hi"); }
#undef PUTC
    const int dstr = ctile >> 2, idx = ((ctile & 3) * 2 + m) * 4 + q;
    char* out_ = myws + WS_STG + ((size_t)((par * 8 + dstr) * 32 + idx) * 4 + wid) * CHUNK;
    const unsigned units = (272 + 2 * total + 15) >> 4;
    asm volatile("s_waitcnt lgkmcnt(0)" ::: "memory");

    {
      u32x4 cp[5];
#pragma unroll
      for (int k = 0; k < 5; ++k) if (k * 64 < units) cp[k] = *(const LAS u32x4*)(size_t)(sb_ + min(k * 64 + lane, CHUNK / 16 - 1) * 16);
#pragma unroll
      for (int k = 0; k < 5; ++k) if (k * 64 + lane < units) st_wt128(out_ + (k * 64 + lane) * 16, cp[k]);
    }
    if (lane == 0) {
      unsigned old;
      asm volatile("ds_max_u32 %1, %2\n\tds_add_rtn_u32 %0, %3, %4\n\ts_waitcnt lgkmcnt(0)" : "=v"(old) : "v"(miscA + 4 * 4), "v"(__float_as_uint(mx)), "v"(miscA + 5 * 4), "v"(1u) : "memory");
      if (old == 3) st_sys64(myws + WS_LMAX + par * 2048 + b * 8, ((unsigned long long)call << 32) | misc[4]);
    }

    char* tfl = p.ws[dstr] + WS_TFL + ((size_t)((par * 8 + p.rank) * 32 + idx) * 4 + wid) * 8;
    asm volatile("" : "+s"(tfl));

    WAIT_V(0);
    if (lane == 0) { __hip_atomic_store((__attribute__((address_space(1))) unsigned long long*)tfl, ((unsigned long long)call << 32) | units, __ATOMIC_RELAXED, __HIP_MEMORY_SCOPE_SYSTEM);
      asm volatile("ds_add_u32 %0, %1" ::"v"(miscA + 27 * 4), "v"(1u) : "memory"); }
  } else {
    const int pw = wid - 4, srcr = (b & 7), idx = (b >> 3);
    const char* in_ = p.ws[srcr] + WS_STG + ((size_t)((par * 8 + p.rank) * 32 + idx) * 4 + pw) * CHUNK;
    const unsigned rb_ = ldsb + 32768 + pw * CHUNK;
    float val[32]; float cscale = 0.f;
    signed char* ob = p.myout + (size_t)(srcr * MS + ((idx >> 2) & 1) * 256 + pw * 64 + (idx & 3) * 16 + (lane & 15)) * NS + (idx >> 3) * 128 + (lane >> 4) * 16;
    asm volatile("" : "+v"(ob));
    bool gdone = !(b == p.rank && wid == 7), tdone = false, adone = wid != 4;
    while (lds_get(miscA + 27 * 4) < 4) __builtin_amdgcn_s_sleep(4);
    bool seen = false;
    bool bad = false;
    auto progress = [&]() {
      unsigned long long gv[4], am = 0, tf = 0;
      unsigned tm = 0;
      const bool pollt = wid == 4 && !seen;
      if (!gdone) tm = ld_sys32(myws + WS_TMO);
      if (pollt) tf = ld_sys64(myws + WS_TFL + ((size_t)((par * 8 + srcr) * 32 + idx) * 4 + (lane & 3)) * 8);
      if (!adone) am = ld_sys64(myws + WS_GMAX + par * 1024 + (b & 7) * 128 + (lane & 15) * 8);
      if (!gdone) {
#pragma unroll
        for (int i = 0; i < 4; ++i) gv[i] = ld_sys64(myws + WS_LMAX + par * 2048 + (i * 64 + lane) * 8);
      }
      bad |= !gdone && tm == call;
      if (!gdone) {
        bool ok = true; unsigned lm = 0;
#pragma unroll
        for (int i = 0; i < 4; ++i) { ok &= (unsigned)(gv[i] >> 32) == call; lm = max(lm, (unsigned)gv[i]); }
        if (__all(ok)) {
          lm = wave_max_u32(lm);
          if (__builtin_amdgcn_readfirstlane(bad)) lm = 0xffffffffu;
          st_sys64(p.ws[lane >> 3] + WS_GMAX + par * 1024 + (lane & 7) * 128 + p.rank * 8, ((unsigned long long)call << 32) | lm);
          gdone = true;
        }
      }
      if (!adone && __all((lane & 8) || (unsigned)(am >> 32) == call)) {
        const bool tmo_ = (unsigned)__builtin_amdgcn_readlane((unsigned)am, 8) == call;
        lds_put(miscA + 1 * 4, wave_max_u32((lane & 8) ? 0u : (unsigned)am)); lds_put(miscA + 2 * 4, tmo_ ? 1u : 0u); lds_put(miscA + 26 * 4, 1); adone = true;
      }
      if (pollt && __all((unsigned)(tf >> 32) == call)) {
        if (lane < 4) *(volatile LAS unsigned*)(size_t)(miscA + (32 + lane) * 4) = (unsigned)tf;
        lds_put(miscA + 28 * 4, 1);
      }
      if (!seen) seen = lds_get(miscA + 28 * 4) != 0;
      if (seen && !tdone && gdone) {
        {
          tf = lds_get(miscA + (32 + pw) * 4);

          const unsigned units = min((unsigned)__builtin_amdgcn_readfirstlane((unsigned)tf), (unsigned)(CHUNK / 16));
          u32x4 r[5];
#pragma unroll
          for (int k = 0; k < 5; ++k) { r[k] = (u32x4){0u, 0u, 0u, 0u}; if (k * 64 + lane < units) asm volatile("global_load_dwordx4 %0, %1, off sc0 sc1" : "+v"(r[k]) : "v"(in_ + (k * 64 + lane) * 16) : "memory"); }
          asm volatile("s_waitcnt vmcnt(0)" : "+v"(r[0]), "+v"(r[1]), "+v"(r[2]), "+v"(r[3]), "+v"(r[4]) :: "memory");
          u32x2 am2 = {0u, 0u};
          if (!adone) asm volatile("global_load_dwordx2 %0, %1, off sc0 sc1" : "+v"(am2) : "v"(myws + WS_GMAX + par * 1024 + (b & 7) * 128 + (lane & 15) * 8) : "memory");

#pragma unroll
          for (int k = 0; k < 5; ++k) if (k * 64 < units) *(LAS u32x4*)(size_t)(rb_ + min(k * 64 + lane, CHUNK / 16 - 1) * 16) = r[k];
          asm volatile("s_waitcnt lgkmcnt(0)" ::: "memory");
          const unsigned mask = *(const LAS unsigned*)(size_t)(rb_ + 16 + lane * 4);
          cscale = __uint_as_float(*(const LAS unsigned*)(size_t)(rb_ + 4));
          bad |= *(const LAS unsigned*)(size_t)(rb_ + 8) != 0u;
          const unsigned cnt = __builtin_popcount(mask);
          unsigned off = rb_ + 272 + 2 * (wave_scan_u32(cnt) - cnt);
          unsigned mm = __builtin_bitreverse32(mask);
          unsigned cv[32];
#pragma unroll
          for (int i = 0; i < 32; ++i) { cv[i] = 0;
            asm volatile("v_add_co_u32 %2, vcc, %2, %2\n\ts_nop 0\n\ts_mov_b64 exec, vcc\n\tds_read_u16 %0, %1\n\tv_add_u32 %1, 2, %1\n\ts_mov_b64 exec, -1" : "+v"(cv[i]), "+v"(off), "+v"(mm) :: "vcc", "memory"); }
          asm volatile("s_waitcnt lgkmcnt(0)" : "+v"(cv[0]), "+v"(cv[1]), "+v"(cv[2]), "+v"(cv[3]), "+v"(cv[4]), "+v"(cv[5]), "+v"(cv[6]), "+v"(cv[7]), "+v"(cv[8]), "+v"(cv[9]), "+v"(cv[10]), "+v"(cv[11]), "+v"(cv[12]), "+v"(cv[13]), "+v"(cv[14]), "+v"(cv[15]) :: "memory");
          asm volatile("" : "+v"(cv[16]), "+v"(cv[17]), "+v"(cv[18]), "+v"(cv[19]), "+v"(cv[20]), "+v"(cv[21]), "+v"(cv[22]), "+v"(cv[23]), "+v"(cv[24]), "+v"(cv[25]), "+v"(cv[26]), "+v"(cv[27]), "+v"(cv[28]), "+v"(cv[29]), "+v"(cv[30]), "+v"(cv[31]));
#pragma unroll
          for (int i = 0; i < 32; ++i) val[i] = (float)cv[i];
          asm volatile("s_waitcnt vmcnt(0)" : "+v"(am2) :: "memory");
          if (!adone && __all((lane & 8) || am2[1] == call)) {
            const bool tmo_ = (unsigned)__builtin_amdgcn_readlane(am2[0], 8) == call;
            lds_put(miscA + 1 * 4, wave_max_u32((lane & 8) ? 0u : am2[0])); lds_put(miscA + 2 * 4, tmo_ ? 1u : 0u); lds_put(miscA + 26 * 4, 1); adone = true;
          }
          tdone = true;
        }
      }
      return gdone && tdone && adone; };
    SPIN_UNTIL(progress());

    if (wid == 4) { if (!adone) { lds_put(miscA + 2 * 4, 1); lds_put(miscA + 26 * 4, 1); } }
    else { while (lds_get(miscA + 26 * 4) == 0) __builtin_amdgcn_s_sleep(1); }
    u32x4 mw; asm volatile("ds_read_b128 %0, %1\n\ts_waitcnt lgkmcnt(0)" : "=v"(mw) : "v"(miscA) : "memory");
    const unsigned amaxb = __builtin_amdgcn_readfirstlane(mw[1]);
    const float amax = __uint_as_float(amaxb);
    bad = __builtin_amdgcn_readfirstlane(bad) || dead || amaxb == 0xffffffffu || __builtin_amdgcn_readfirstlane(mw[2]) != 0;

    const float cq = amax > 0.f ? cscale * (127.0f / 65535.0f) / amax : 0.f;
    unsigned d[8];
#pragma unroll
    for (int nb = 0; nb < 8; ++nb) {
      unsigned v = 0;
#pragma unroll
      for (int e = 0; e < 4; ++e) v = __builtin_amdgcn_cvt_pk_u8_f32(fminf(val[nb * 4 + e] * cq, 127.0f), e, v);
      d[nb] = bad ? 0x80808080u : v;
    }
#pragma unroll
    for (int hh = 0; hh < 2; ++hh) {
      const auto s02 = __builtin_amdgcn_permlane32_swap(d[hh * 4 + 0], d[hh * 4 + 2], false, false);
      const auto s13 = __builtin_amdgcn_permlane32_swap(d[hh * 4 + 1], d[hh * 4 + 3], false, false);
      const auto t01 = __builtin_amdgcn_permlane16_swap(s02[0], s13[0], false, false);
      const auto t23 = __builtin_amdgcn_permlane16_swap(s02[1], s13[1], false, false);
      *(__attribute__((address_space(1))) u32x4*)(ob + hh * 64) = (u32x4){t01[0], t01[1], t23[0], t23[1]};
    }
  }
}

extern "C" void kernel_launch(int rank, int nranks, ncclComm_t comm, hipStream_t stream,
                              void* const* d_in, const int* in_sizes, int n_in,
                              void* d_out, int out_size, const int* in_dtypes, int out_dtype,
                              void* const* d_out_all, void* const* d_ws_all, size_t ws_bytes) {
  if (nranks != NR || ws_bytes < WS_END || n_in < 2) { fprintf(stderr, "bad config\n"); return; }
  if (in_sizes[0] != MS * KD || in_sizes[1] != KD * ND || out_size != 4096 * NS) { fprintf(stderr, "bad sizes\n"); return; }
  HIP_CHECK(hipFuncSetAttribute((const void*)gemm_a2a, hipFuncAttributeMaxDynamicSharedMemorySize, LDS_BYTES));
  Params p;
  p.x = (const float*)d_in[0]; p.w = (const float*)d_in[1];
  for (int i = 0; i < NR; ++i) p.ws[i] = (char*)d_ws_all[i];
  p.myws = (char*)d_ws_all[rank]; p.myout = (signed char*)d_out;
  p.rank = rank; p.pad = 0;
  hipLaunchKernelGGL(gemm_a2a, dim3(NWG), dim3(NTHR), LDS_BYTES, stream, p);
  HIP_CHECK(hipGetLastError());
}
```
